```python
import numpy as np

kernel_name = "gpu_ag_gemm_m4096_k4096_n16384_w8a8_dequant"


ABSMAX_RTOL = 0.001

COMPUTE_DTYPE = "int8"
EPILOGUE = "dequant"
IN_QUANT = "int8"
OUT_QUANT = None
IN_DTYPES = {"x": "int8", "w_mat": "int8",
             "scale_x": "float32", "scale_w": "float32"}
WORLD_SIZE = 8

def setup_inputs():

    x = np.random.randint(-127, 128, size=(4096, 4096)).astype(np.int8)
    w_mat = np.random.randint(-127, 128, size=(4096, 16384)).astype(np.int8)
    scale_x = (0.02 * (1.0 + np.random.rand(1))).astype(np.float32)
    scale_w = (0.02 * (1.0 + np.random.rand(1))).astype(np.float32)
    return {"x": x, "w_mat": w_mat, "scale_x": scale_x, "scale_w": scale_w}

def reference(x, w_mat, scale_x, scale_w):
    acc = x.astype(np.float64) @ w_mat.astype(np.float64)
    y = (acc.astype(np.float32) * (scale_x * scale_w)).astype(np.float32)
    return y
```

```cpp
#include <hip/hip_runtime.h>
#include <rccl/rccl.h>
#include <cstdio>
#include <cstdint>
#include <utility>

#define HIP_CHECK(expr) do { hipError_t _e = (expr); if (_e != hipSuccess) { \
    fprintf(stderr, "HIP error %s at %s:%d\n", hipGetErrorString(_e), __FILE__, __LINE__); \
    return; } } while (0)

namespace agk {

typedef int i32x4 __attribute__((ext_vector_type(4)));
typedef unsigned u32x4 __attribute__((ext_vector_type(4)));
typedef float f32x4 __attribute__((ext_vector_type(4)));
#define LAS __attribute__((address_space(3)))

constexpr int NR = 8;
constexpr int KDIM = 4096, MSH = 512, NSH = 2048;
constexpr int NRB = MSH / 16;
constexpr unsigned RB_BYTES = 16u * KDIM;
constexpr unsigned SHARD_BYTES = (unsigned)MSH * KDIM;
constexpr unsigned W_BYTES = (unsigned)KDIM * NSH;
constexpr unsigned OUT_BYTES = (unsigned)NR * MSH * NSH * 4u;

constexpr size_t CTR_OFF = 0;
constexpr size_t TMO_OFF = 16384;
constexpr size_t FLAG_OFF = 32768;
constexpr size_t STAMP_OFF = 65536;
constexpr size_t STAGE_OFF = 1u << 20;
constexpr size_t WS_NEED = STAGE_OFF + 2ull * NR * SHARD_BYTES;

constexpr unsigned long long DEADLINE_TICKS = 50000000ull;

constexpr int NS = 4;
constexpr int RING_OFF = 0;
constexpr int RED_OFF = 8 * 2 * 8192;
constexpr int CTL_OFF = RED_OFF + NS * 4096;
constexpr int POLL_OFF = CTL_OFF + 256;
constexpr int LDS_BYTES = POLL_OFF + 8 * 256;

constexpr int AUX_SYS = 17;
#define AGK_XAUX 17
#define AGK_OAUX 0
#define AGK_WIN 8
#define AGK_LAG 1

struct Params {
  const char* x;
  const char* w;
  const float* sx;
  const float* sw;
  float* out;
  char* ws[NR];
  int rank;
};

template <int N>
__device__ __forceinline__ void wait_vm() {
  static_assert(N >= 0 && N < 64, "vmcnt range");
  asm volatile("s_waitcnt vmcnt(%0)" ::"n"(N) : "memory");
}
__device__ __forceinline__ void wait_lgkm0() { asm volatile("s_waitcnt lgkmcnt(0)" ::: "memory"); }
__device__ __forceinline__ void cfence() { asm volatile("" ::: "memory"); }

__device__ __forceinline__ u32x4 make_srd(const void* base, unsigned bytes) {
  const unsigned long long a = (unsigned long long)base;
  u32x4 r;
  r.x = (unsigned)__builtin_amdgcn_readfirstlane((unsigned)a);
  r.y = (unsigned)__builtin_amdgcn_readfirstlane((unsigned)(a >> 32) & 0xffffu);
  r.z = bytes;
  r.w = 0x00020000u;
  return r;
}
#define AGK_XPOL " sc0 sc1"
__device__ __forceinline__ void dma_tiled(u32x4 srd, unsigned lds, unsigned voff, unsigned soff) {
  asm volatile(
      "s_mov_b32 m0, %0\n\ts_nop 3\n\t"
      "buffer_load_dwordx4 %2, %3, %4 offen" AGK_XPOL " lds\n\t"
      "buffer_load_dwordx4 %2, %3, %4 offen offset:1024" AGK_XPOL " lds\n\t"
      "buffer_load_dwordx4 %2, %3, %4 offen offset:2048" AGK_XPOL " lds\n\t"
      "buffer_load_dwordx4 %2, %3, %4 offen offset:3072" AGK_XPOL " lds\n\t"
      "s_mov_b32 m0, %1\n\ts_nop 0\n\t"
      "buffer_load_dwordx4 %2, %3, %5 offen" AGK_XPOL " lds\n\t"
      "buffer_load_dwordx4 %2, %3, %5 offen offset:1024" AGK_XPOL " lds\n\t"
      "buffer_load_dwordx4 %2, %3, %5 offen offset:2048" AGK_XPOL " lds\n\t"
      "buffer_load_dwordx4 %2, %3, %5 offen offset:3072" AGK_XPOL " lds"
      ::"s"(lds), "s"(lds + 4096u), "v"(voff), "s"(srd), "s"(soff), "s"(soff + 4096u)
      : "memory");
}
__device__ __forceinline__ void dma_rows(u32x4 srd, unsigned lds, unsigned voff, unsigned soff) {
  asm volatile(
      "s_mov_b32 m0, %0\n\ts_nop 3\n\t"
      "buffer_load_dwordx4 %1, %2, %3 offen lds\n\t"
      "s_add_u32 m0, m0, 960\n\ts_nop 0\n\t"
      "buffer_load_dwordx4 %1, %2, %3 offen offset:64 lds\n\t"
      "s_add_u32 m0, m0, 960\n\ts_nop 0\n\t"
      "buffer_load_dwordx4 %1, %2, %3 offen offset:128 lds\n\t"
      "s_add_u32 m0, m0, 960\n\ts_nop 0\n\t"
      "buffer_load_dwordx4 %1, %2, %3 offen offset:192 lds\n\t"
      "s_add_u32 m0, m0, 960\n\ts_nop 0\n\t"
      "buffer_load_dwordx4 %1, %2, %3 offen offset:256 lds\n\t"
      "s_add_u32 m0, m0, 960\n\ts_nop 0\n\t"
      "buffer_load_dwordx4 %1, %2, %3 offen offset:320 lds\n\t"
      "s_add_u32 m0, m0, 960\n\ts_nop 0\n\t"
      "buffer_load_dwordx4 %1, %2, %3 offen offset:384 lds\n\t"
      "s_add_u32 m0, m0, 960\n\ts_nop 0\n\t"
      "buffer_load_dwordx4 %1, %2, %3 offen offset:448 lds"
      ::"s"(lds), "v"(voff), "s"(srd), "s"(soff)
      : "memory", "scc");
}

__device__ __forceinline__ void dma_word(u32x4 srd, unsigned lds, unsigned voff) {
  asm volatile("s_mov_b32 m0, %0\n\ts_nop 3\n\tbuffer_load_dword %1, %2, 0 offen sc0 sc1 lds"
               ::"s"(lds), "v"(voff), "s"(srd) : "memory");
}

#define STAMP(idx) do { } while (0)

struct CommCtx {
  __amdgpu_buffer_rsrc_t xr, dr;
  unsigned xoff, doff;
  unsigned* flag;
  LAS unsigned* cnt;
  unsigned call;
  int lane;
};

constexpr int WIN = AGK_WIN, LAG = AGK_LAG;
constexpr int loads_after(int u) { return (u + WIN < NRB) ? 2 : 0; }
constexpr int ops_after(int r, int rb) {
  int n = 0;
  for (int u = r; u <= rb; ++u) n += loads_after(u);
  return n + 2 * (rb - r);
}

__device__ __forceinline__ void comm_signal(const CommCtx& c, int r) {
  if (c.lane == 0) {
    unsigned old = __hip_atomic_fetch_add(c.cnt + r, 1u, __ATOMIC_RELAXED, __HIP_MEMORY_SCOPE_WORKGROUP);
    if (old == 7u) __hip_atomic_store(c.flag + r, c.call, __ATOMIC_RELAXED, __HIP_MEMORY_SCOPE_SYSTEM);
  }
  cfence();
}

template <int RB>
__device__ __forceinline__ void comm_step(const CommCtx& c, u32x4 (&v)[WIN][2]) {
  __builtin_amdgcn_raw_buffer_store_b128(v[RB % WIN][0], c.dr, c.doff, RB * RB_BYTES, AUX_SYS);
  __builtin_amdgcn_raw_buffer_store_b128(v[RB % WIN][1], c.dr, c.doff + 1024, RB * RB_BYTES, AUX_SYS);
  cfence();
  if constexpr (RB + WIN < NRB) {
    v[RB % WIN][0] = __builtin_amdgcn_raw_buffer_load_b128(c.xr, c.xoff, (RB + WIN) * RB_BYTES, 0);
    v[RB % WIN][1] = __builtin_amdgcn_raw_buffer_load_b128(c.xr, c.xoff + 64, (RB + WIN) * RB_BYTES, 0);
    cfence();
  }
  if constexpr (RB >= LAG) {
    wait_vm<ops_after(RB - LAG, RB)>();
    comm_signal(c, RB - LAG);
  }
}
template <int R>
__device__ __forceinline__ void comm_tail(const CommCtx& c) {
  wait_vm<ops_after(R, NRB - 1)>();
  comm_signal(c, R);
}
template <int... I>
__device__ __forceinline__ void comm_steps(const CommCtx& c, u32x4 (&v)[WIN][2], std::integer_sequence<int, I...>) {
  (comm_step<I>(c, v), ...);
}
template <int... I>
__device__ __forceinline__ void comm_tails(const CommCtx& c, std::integer_sequence<int, I...>) {
  (comm_tail<NRB - LAG + I>(c), ...);
}

__global__ void __launch_bounds__(512, 2) ag_gemm_kernel(Params p) {
  __shared__ __attribute__((aligned(16))) char smem_raw[LDS_BYTES];
  LAS char* smem = (LAS char*)smem_raw;
  const int b = blockIdx.x, xcd = b & 7, slot = b >> 3;
  const int wave = __builtin_amdgcn_readfirstlane(threadIdx.x >> 6), lane = threadIdx.x & 63;
  const int rank = p.rank & (NR - 1);
  char* my = p.ws[rank];
  unsigned* ctr = (unsigned*)(my + CTR_OFF + (size_t)b * 64);
  LAS unsigned* ctl = (LAS unsigned*)(smem + CTL_OFF);
  STAMP(0);
  const unsigned ctr_val = __hip_atomic_load(ctr, __ATOMIC_RELAXED, __HIP_MEMORY_SCOPE_AGENT);
  cfence();
  const __amdgpu_buffer_rsrc_t xr = __builtin_amdgcn_make_buffer_rsrc((void*)p.x, 0, (int)SHARD_BYTES, 0x00020000);

  if (slot < 4) {
    if (xcd == 7) return;
    const int q = slot;
    const unsigned peer = (unsigned)(rank + 1 + xcd) & (NR - 1);
    CommCtx c;
    c.xr = xr;
    c.xoff = (lane & 15) * KDIM + (lane >> 4) * 16 + q * 1024 + wave * 128;
    u32x4 v[WIN][2];
#pragma unroll
    for (int r = 0; r < WIN; ++r) {
      v[r][0] = __builtin_amdgcn_raw_buffer_load_b128(xr, c.xoff, r * RB_BYTES, 0);
      v[r][1] = __builtin_amdgcn_raw_buffer_load_b128(xr, c.xoff + 64, r * RB_BYTES, 0);
    }
    cfence();
    if (threadIdx.x < NRB) ctl[threadIdx.x] = 0u;
    unsigned call = (unsigned)__builtin_amdgcn_readfirstlane(ctr_val) + 1u;
    asm volatile("" : "+s"(call));
    wait_lgkm0();
    __builtin_amdgcn_s_barrier();
    cfence();
    if (threadIdx.x == 0) __hip_atomic_store(ctr, call, __ATOMIC_RELAXED, __HIP_MEMORY_SCOPE_AGENT);
    char* pw = p.ws[peer];
    c.dr = __builtin_amdgcn_make_buffer_rsrc((void*)(pw + STAGE_OFF + (size_t)((call & 1u) * NR + rank) * SHARD_BYTES), 0,
                                             (int)SHARD_BYTES, 0x00020000);
    c.doff = lane * 16 + (q * 16 + wave * 2) * 1024;
    c.flag = (unsigned*)(pw + FLAG_OFF + (size_t)(rank * 4 + q) * 256);
    c.cnt = ctl;
    c.call = call;
    c.lane = lane;
    cfence();
    STAMP(1);
    comm_steps(c, v, std::make_integer_sequence<int, NRB>{});
    STAMP(2);
    comm_tails(c, std::make_integer_sequence<int, LAG>{});
    STAMP(3);
    return;
  }

  const int idx = slot - 4;
  const int panel = xcd * 4 + (idx & 3);
  const int si = idx >> 2;
  const int src = (rank + 1 + si) & (NR - 1);
  const int nloc = si < 4 ? 5 : 4;
  const int ntot = nloc + NRB;
  const int lc = lane & 15, lg = lane >> 4;

  LAS char* ring = smem + RING_OFF + wave * 16384;
  const unsigned voff_l = lc * KDIM + lg * 16 + wave * 512;
  const unsigned voff_s = lane * 16 + wave * 8192;

  const unsigned ring_a = (unsigned)(unsigned long long)ring;
  const u32x4 x_srd = make_srd(p.x, SHARD_BYTES);
  auto issue_local = [&](int j) {
    dma_rows(x_srd, ring_a + (j & 1) * 8192, voff_l, (unsigned)(si + 7 * j) * RB_BYTES);
  };
  issue_local(0);
  issue_local(1);
  cfence();

  i32x4 wf[8][4];
  {
    const __amdgpu_buffer_rsrc_t wr = __builtin_amdgcn_make_buffer_rsrc((void*)p.w, 0, (int)W_BYTES, 0x00020000);
    const unsigned voff_w = (unsigned)(wave * 512 + lg * 16) * NSH + panel * 64 + lc * 4;
    unsigned raw[8][4][4];
#pragma unroll
    for (int t = 0; t < 8; ++t)
#pragma unroll
      for (int d = 0; d < 4; ++d)
#pragma unroll
        for (int i = 0; i < 4; ++i)
          raw[t][d][i] = __builtin_amdgcn_raw_buffer_load_b32(wr, voff_w + (i & 1) * NSH, (64 * t + 4 * d + (i & 2)) * NSH, 0);
    cfence();
    {
      LAS u32x4* red = (LAS u32x4*)(smem + RED_OFF);
#pragma unroll
      for (int u = 0; u < NS * 4096 / 16 / 512; ++u) red[u * 512 + threadIdx.x] = (u32x4){0u, 0u, 0u, 0u};
      if (threadIdx.x < 64) ctl[threadIdx.x] = 0u;
    }
#pragma unroll
    for (int t = 0; t < 8; ++t)
#pragma unroll
      for (int d = 0; d < 4; ++d) {
        const unsigned r0 = raw[t][d][0], r1 = raw[t][d][1], r2 = raw[t][d][2], r3 = raw[t][d][3];
        const unsigned t0 = __builtin_amdgcn_perm(r1, r0, 0x05010400u);
        const unsigned t1 = __builtin_amdgcn_perm(r1, r0, 0x07030602u);
        const unsigned t2 = __builtin_amdgcn_perm(r3, r2, 0x05010400u);
        const unsigned t3 = __builtin_amdgcn_perm(r3, r2, 0x07030602u);
        wf[t][0][d] = (int)__builtin_amdgcn_perm(t2, t0, 0x05040100u);
        wf[t][1][d] = (int)__builtin_amdgcn_perm(t2, t0, 0x07060302u);
        wf[t][2][d] = (int)__builtin_amdgcn_perm(t3, t1, 0x05040100u);
        wf[t][3][d] = (int)__builtin_amdgcn_perm(t3, t1, 0x07060302u);
      }
  }
  const float scale = p.sx[0] * p.sw[0];
  unsigned call = (unsigned)__builtin_amdgcn_readfirstlane(ctr_val) + 1u;
  asm volatile("" : "+s"(call));
  wait_vm<0>();
  wait_lgkm0();
  __builtin_amdgcn_s_barrier();
  cfence();
  STAMP(1);
  if (threadIdx.x == 0) __hip_atomic_store(ctr, call, __ATOMIC_RELAXED, __HIP_MEMORY_SCOPE_AGENT);

  const u32x4 s_srd = make_srd(my + STAGE_OFF + (size_t)((call & 1u) * NR + src) * SHARD_BYTES, SHARD_BYTES);
  const __amdgpu_buffer_rsrc_t orr = __builtin_amdgcn_make_buffer_rsrc((void*)p.out, 0, (int)OUT_BYTES, 0x00020000);
  const unsigned* flag = (const unsigned*)(my + FLAG_OFF + (size_t)(src * 4 + (wave >> 1)) * 256) + (lane & 31);
  unsigned* tmo = (unsigned*)(my + TMO_OFF);
  const u32x4 f_srd = make_srd(my + FLAG_OFF + (size_t)(src * 4 + (wave >> 1)) * 256, 128);
  LAS unsigned* snap = (LAS unsigned*)(smem + POLL_OFF + wave * 256);
  const unsigned snap_a = (unsigned)(unsigned long long)snap;
  LAS unsigned* cnt = ctl;
  LAS unsigned* gen = ctl + 8;
  LAS unsigned* dead = ctl + 16;
  const unsigned voff_o = (unsigned)(lg * 4 * NSH + panel * 64 + lc * 4) * 4u;

  auto issue_remote = [&](int j) {
    dma_tiled(s_srd, ring_a + (j & 1) * 8192, voff_s, (unsigned)(j - nloc) * RB_BYTES);
  };
  auto count_ready = [&](unsigned v) -> int {
    const unsigned m = (unsigned)__ballot((int)(v - call) >= 0);
    return m == 0xffffffffu ? NRB : __builtin_ctz(~m);
  };

  int issued = 2;
  int avail = nloc;
  int after0 = 0, after1 = 0;
  auto note_ops = [&](int n) { after0 += n; after1 += n; };
  auto issue = [&](int j) {
    if (j < nloc) issue_local(j); else issue_remote(j);
    cfence();
    note_ops(8);
    if (j & 1) after1 = 0; else after0 = 0;
  };

  for (int j = 0; j < ntot; ++j) {
    if (issued == j) {
      const unsigned long long t0 = __builtin_amdgcn_s_memrealtime();
      bool gave_up = false;
      for (unsigned spins = 1;; ++spins) {
        const unsigned v = __hip_atomic_load(flag, __ATOMIC_RELAXED, __HIP_MEMORY_SCOPE_SYSTEM);
        avail = nloc + count_ready(v);
        if (avail > j) break;
        if ((spins & 31u) == 0u) {
          const unsigned tv = (unsigned)__builtin_amdgcn_readfirstlane(__hip_atomic_load(tmo, __ATOMIC_RELAXED, __HIP_MEMORY_SCOPE_AGENT));
          if (tv == call || __builtin_amdgcn_s_memrealtime() - t0 > DEADLINE_TICKS) { gave_up = true; break; }
        }
        __builtin_amdgcn_s_sleep(1);
      }
      cfence();
      if (gave_up) {
        __hip_atomic_store(tmo, call, __ATOMIC_RELAXED, __HIP_MEMORY_SCOPE_AGENT);
        __hip_atomic_store(dead, 1u, __ATOMIC_RELAXED, __HIP_MEMORY_SCOPE_WORKGROUP);
        avail = ntot;
      }
      cfence();
      issue(j);
      issued = j + 1;
      if (avail > issued) { issue(issued); ++issued; }
    }
    {
      const int n = (j & 1) ? after1 : after0;
      if (n >= 12) wait_vm<12>();
      else if (n >= 8) wait_vm<8>();
      else if (n >= 4) wait_vm<4>();
      else wait_vm<0>();
    }
    i32x4 xf[8];
    {
      const LAS char* rp = ring + (j & 1) * 8192 + lane * 16;
#pragma unroll
      for (int t = 0; t < 8; ++t) xf[t] = *(const LAS i32x4*)(rp + t * 1024);
    }
    const int rs = j & (NS - 1);
    const unsigned gen_need = (unsigned)(j / NS);
    unsigned gen_v = __hip_atomic_load(gen + rs, __ATOMIC_RELAXED, __HIP_MEMORY_SCOPE_WORKGROUP);
    wait_lgkm0();
    asm volatile("" : "+v"(xf[0]), "+v"(xf[1]), "+v"(xf[2]), "+v"(xf[3]), "+v"(xf[4]), "+v"(xf[5]), "+v"(xf[6]), "+v"(xf[7]));
    {
      const int lim = (avail < j + 3) ? avail : j + 3;
      while (issued < lim) { issue(issued); ++issued; }
    }
    const bool do_poll = (issued < j + 3) && (issued < ntot);
    if (do_poll) {
      dma_word(f_srd, snap_a, (lane & 31) * 4);
      note_ops(1);
    }
    i32x4 acc[4];
#pragma unroll
    for (int g = 0; g < 4; ++g) acc[g] = (i32x4){0, 0, 0, 0};
#pragma unroll
    for (int t = 0; t < 8; ++t)
#pragma unroll
      for (int g = 0; g < 4; ++g) acc[g] = __builtin_amdgcn_mfma_i32_16x16x64_i8(xf[t], wf[t][g], acc[g], 0, 0, 0);

    while ((unsigned)__builtin_amdgcn_readfirstlane(gen_v) < gen_need) {
      __builtin_amdgcn_s_sleep(1);
      gen_v = __hip_atomic_load(gen + rs, __ATOMIC_RELAXED, __HIP_MEMORY_SCOPE_WORKGROUP);
    }
    cfence();
    LAS int* red = (LAS int*)(smem + RED_OFF + rs * 4096) + lane;
#pragma unroll
    for (int r = 0; r < 4; ++r)
#pragma unroll
      for (int g = 0; g < 4; ++g)
        __hip_atomic_fetch_add(red + (r * 4 + g) * 64, acc[g][r], __ATOMIC_RELAXED, __HIP_MEMORY_SCOPE_WORKGROUP);
    wait_lgkm0();
    unsigned old = 0u;
    if (lane == 0) old = __hip_atomic_fetch_add(cnt + rs, 1u, __ATOMIC_RELAXED, __HIP_MEMORY_SCOPE_WORKGROUP);
    old = (unsigned)__builtin_amdgcn_readfirstlane(old);
    cfence();
    const bool last = (old & 7u) == 7u;
    if (last) {
      int s[4][4];
#pragma unroll
      for (int r = 0; r < 4; ++r)
#pragma unroll
        for (int g = 0; g < 4; ++g) s[r][g] = __hip_atomic_load(red + (r * 4 + g) * 64, __ATOMIC_RELAXED, __HIP_MEMORY_SCOPE_WORKGROUP);
      const unsigned dv = __hip_atomic_load(dead, __ATOMIC_RELAXED, __HIP_MEMORY_SCOPE_WORKGROUP);
      wait_lgkm0();
      LAS u32x4* z = (LAS u32x4*)(smem + RED_OFF + rs * 4096) + lane;
#pragma unroll
      for (int u = 0; u < 4; ++u) z[u * 64] = (u32x4){0u, 0u, 0u, 0u};
      wait_lgkm0();
      if (lane == 0) __hip_atomic_store(gen + rs, gen_need + 1u, __ATOMIC_RELAXED, __HIP_MEMORY_SCOPE_WORKGROUP);
      const float sc = dv ? __builtin_nanf("") : scale;
      const int row0 = (j < nloc) ? rank * MSH + (si + 7 * j) * 16 : src * MSH + (j - nloc) * 16;
      const unsigned so = (unsigned)row0 * (NSH * 4u);
#pragma unroll
      for (int r = 0; r < 4; ++r) {
        f32x4 y;
#pragma unroll
        for (int g = 0; g < 4; ++g) y[g] = (float)s[r][g] * sc;
        __builtin_amdgcn_raw_buffer_store_b128(__builtin_bit_cast(u32x4, y), orr, voff_o + so + r * (NSH * 4u), 0, AGK_OAUX);
      }
      cfence();
      note_ops(4);
    }
    if (do_poll) {
      if (last) wait_vm<4>(); else wait_vm<0>();
      const unsigned pv = __hip_atomic_load(snap + lane, __ATOMIC_RELAXED, __HIP_MEMORY_SCOPE_WORKGROUP);
      const int a = nloc + count_ready(pv);
      avail = a > avail ? a : avail;
      cfence();
      const int lim = (avail < j + 3) ? avail : j + 3;
      while (issued < lim) { issue(issued); ++issued; }
    }
  }
}

}

extern "C" void kernel_launch(int rank, int nranks, ncclComm_t comm,
                              hipStream_t stream,
                              void* const* d_in, const int* in_sizes, int n_in,
                              void* d_out, int out_size,
                              const int* in_dtypes, int out_dtype,
                              void* const* d_out_all,
                              void* const* d_ws_all, size_t ws_bytes) {
  (void)comm; (void)in_dtypes; (void)out_dtype; (void)d_out_all;
  if (nranks != agk::NR || (unsigned)rank >= (unsigned)agk::NR || n_in < 4 || ws_bytes < agk::WS_NEED ||
      in_sizes[0] != agk::MSH * agk::KDIM || in_sizes[1] != agk::KDIM * agk::NSH ||
      out_size != agk::NR * agk::MSH * agk::NSH) {
    fprintf(stderr, "kernel_launch: unexpected problem shape\n");
    return;
  }
  agk::Params p;
  p.x = (const char*)d_in[0];
  p.w = (const char*)d_in[1];
  p.sx = (const float*)d_in[2];
  p.sw = (const float*)d_in[3];
  p.out = (float*)d_out;
  for (int r = 0; r < agk::NR; ++r) p.ws[r] = (char*)d_ws_all[r];
  p.rank = rank;
  hipLaunchKernelGGL(agk::ag_gemm_kernel, dim3(256), dim3(512), 0, stream, p);
  HIP_CHECK(hipGetLastError());
}
```

```cpp
#include <hip/hip_runtime.h>
#include <rccl/rccl.h>
#include <cstdio>
#include <cstdint>
#include <utility>

#define HIP_CHECK(expr) do { hipError_t _e = (expr); if (_e != hipSuccess) { \
    fprintf(stderr, "HIP error %s at %s:%d\n", hipGetErrorString(_e), __FILE__, __LINE__); \
    return; } } while (0)

namespace agk {

typedef int i32x4 __attribute__((ext_vector_type(4)));
typedef unsigned u32x4 __attribute__((ext_vector_type(4)));
typedef float f32x4 __attribute__((ext_vector_type(4)));
#define LAS __attribute__((address_space(3)))

constexpr int NR = 8;
constexpr int KDIM = 4096, MSH = 512, NSH = 2048;
constexpr int NRB = MSH / 16;
constexpr unsigned RB_BYTES = 16u * KDIM;
constexpr unsigned SHARD_BYTES = (unsigned)MSH * KDIM;
constexpr unsigned W_BYTES = (unsigned)KDIM * NSH;
constexpr unsigned OUT_BYTES = (unsigned)NR * MSH * NSH * 4u;

constexpr size_t CTR_OFF = 0;
constexpr size_t TMO_OFF = 16384;
constexpr size_t FLAG_OFF = 32768;
constexpr size_t STAMP_OFF = 65536;
constexpr size_t STAGE_OFF = 1u << 20;
constexpr size_t WS_NEED = STAGE_OFF + 2ull * NR * SHARD_BYTES;

constexpr unsigned long long DEADLINE_TICKS = 50000000ull;

#define AGK_RLAG 1
constexpr int RLAG = AGK_RLAG;
constexpr int NS = 2 * RLAG;
constexpr int PBLK = 272;
constexpr int PWAVE = 16 * PBLK;
constexpr int PART_BYTES = 8 * PWAVE;
constexpr int WSTAGE = 16384;
constexpr int CTL_OFF = 140 * 1024;
constexpr int LDS_BYTES = CTL_OFF + 256;
static_assert(NS * PART_BYTES <= CTL_OFF && 8 * WSTAGE <= CTL_OFF, "LDS map");

constexpr int AUX_SYS = 17;
#define AGK_XAUX 17
#define AGK_OAUX 0
#define AGK_WIN 8
#define AGK_LAG 1
#define AGK_ROT 1
#define AGK_WAUX 0
#define AGK_SELFAUX 16
#define AGK_ORDER 1
#define AGK_PRIO 0
#define AGK_WSWZ 1
#define AGK_RT 5
#define AGK_MAP 2
#define AGK_WDELAY 0

struct Params {
  const char* x;
  const char* w;
  const float* sx;
  const float* sw;
  float* out;
  char* my;
  char* peer_ws[NR];
  int rank;
};

template <int N>
__device__ __forceinline__ void wait_vm() {
  static_assert(N >= 0 && N < 64, "vmcnt range");
  asm volatile("s_waitcnt vmcnt(%0)" ::"n"(N) : "memory");
}
__device__ __forceinline__ void wait_lgkm0() { asm volatile("s_waitcnt lgkmcnt(0)" ::: "memory"); }
__device__ __forceinline__ void cfence() { asm volatile("" ::: "memory"); }
__device__ __forceinline__ void lds_wait_ge(unsigned a, unsigned need) {
  unsigned tv, ts;
  asm volatile(
      "1:\n\t"
      "ds_read_b32 %0, %2\n\t"
      "s_waitcnt lgkmcnt(0)\n\t"
      "v_readfirstlane_b32 %1, %0\n\t"
      "s_cmp_lt_u32 %1, %3\n\t"
      "s_cbranch_scc1 1b"
      : "=&v"(tv), "=&s"(ts) : "v"(a), "s"(need) : "scc", "memory");
}

__device__ __forceinline__ void lds_inc(unsigned a) {
  asm volatile("s_mov_b64 exec, 1\n\tds_add_u32 %0, %1\n\ts_mov_b64 exec, -1" ::"v"(a), "v"(1u) : "memory");
}
__device__ __forceinline__ u32x4 make_srd(const void* base, unsigned bytes) {
  const unsigned long long a = (unsigned long long)base;
  u32x4 r;
  r.x = (unsigned)__builtin_amdgcn_readfirstlane((unsigned)a);
  r.y = (unsigned)__builtin_amdgcn_readfirstlane((unsigned)(a >> 32) & 0xffffu);
  r.z = bytes;
  r.w = 0x00020000u;
  return r;
}
__device__ __forceinline__ void dma_w(u32x4 srd, unsigned lds, unsigned v_even, unsigned v_odd, unsigned soff) {
  asm volatile(
      "s_mov_b32 m0, %0\n\ts_nop 3\n\t"
      "buffer_load_dwordx4 %1, %3, %4 offen lds\n\t"
      "s_add_u32 m0, m0, 1024\n\ts_nop 0\n\t"
      "buffer_load_dwordx4 %2, %3, %4 offen lds\n\t"
      "s_add_u32 m0, m0, 1024\n\ts_nop 0\n\t"
      "buffer_load_dwordx4 %1, %3, %5 offen lds\n\t"
      "s_add_u32 m0, m0, 1024\n\ts_nop 0\n\t"
      "buffer_load_dwordx4 %2, %3, %5 offen lds"
      ::"s"(lds), "v"(v_even), "v"(v_odd), "s"(srd), "s"(soff), "s"(soff + 32u * NSH)
      : "memory", "scc");
}

#define STAMP(idx) do { } while (0)

struct CommCtx {
  __amdgpu_buffer_rsrc_t xr, dr;
  unsigned xoff, doff;
  unsigned* flag;
  unsigned* ctr;
  unsigned call;
  int wave;
  bool self;
};

constexpr int WIN = AGK_WIN, LAG = AGK_LAG;
constexpr int INIT = 2;
constexpr int loaded_by(int u) {
  int a = INIT + 2 * (u + 1), b = u + 1 + WIN;
  a = a < b ? a : b;
  return a < NRB ? a : NRB;
}
constexpr int loads_after(int u) { return 2 * (loaded_by(u) - loaded_by(u - 1)); }
constexpr int ops_after(int r, int rb) {
  int n = 0;
  for (int u = r; u <= rb; ++u) n += loads_after(u);
  return n + 2 * (rb - r);
}
__device__ __forceinline__ unsigned comm_strip(const CommCtx& c, int rb) { return (unsigned)((c.wave + AGK_ROT * rb) & 7); }

__device__ __forceinline__ void comm_load(const CommCtx& c, int rb, u32x4 (&v)[2]) {
  const unsigned so = rb * RB_BYTES + comm_strip(c, rb) * 128u;
  v[0] = __builtin_amdgcn_raw_buffer_load_b128(c.xr, c.xoff, so, 0);
  v[1] = __builtin_amdgcn_raw_buffer_load_b128(c.xr, c.xoff + 64, so, 0);
}
template <int R>
__device__ __forceinline__ void comm_signal(const CommCtx& c) {
  __builtin_amdgcn_s_barrier();
  cfence();
  if (threadIdx.x == (R & 7) * 64) {
    __hip_atomic_store(c.flag + R, c.call, __ATOMIC_RELAXED, __HIP_MEMORY_SCOPE_SYSTEM);
    if (R == 0) __hip_atomic_store(c.ctr, c.call, __ATOMIC_RELAXED, __HIP_MEMORY_SCOPE_AGENT);
  }
  cfence();
}

template <int RB>
__device__ __forceinline__ void comm_step(const CommCtx& c, u32x4 (&v)[WIN][2]) {
  const unsigned so = RB * RB_BYTES + comm_strip(c, RB) * 2048u;
  if (c.self) {
    __builtin_amdgcn_raw_buffer_store_b128(v[RB % WIN][0], c.dr, c.doff, so, AGK_SELFAUX);
    __builtin_amdgcn_raw_buffer_store_b128(v[RB % WIN][1], c.dr, c.doff + 1024, so, AGK_SELFAUX);
  } else {
    __builtin_amdgcn_raw_buffer_store_b128(v[RB % WIN][0], c.dr, c.doff, so, AUX_SYS);
    __builtin_amdgcn_raw_buffer_store_b128(v[RB % WIN][1], c.dr, c.doff + 1024, so, AUX_SYS);
  }
  cfence();
#pragma unroll
  for (int r = loaded_by(RB - 1); r < loaded_by(RB); ++r) comm_load(c, r, v[r % WIN]);
  cfence();
  if constexpr (RB >= LAG) {
    wait_vm<ops_after(RB - LAG, RB)>();
    comm_signal<RB - LAG>(c);
  }
}
template <int R>
__device__ __forceinline__ void comm_tail(const CommCtx& c) {
  wait_vm<ops_after(R, NRB - 1)>();
  comm_signal<R>(c);
}
template <int... I>
__device__ __forceinline__ void comm_steps(const CommCtx& c, u32x4 (&v)[WIN][2], std::integer_sequence<int, I...>) {
  (comm_step<I>(c, v), ...);
}
template <int... I>
__device__ __forceinline__ void comm_tails(const CommCtx& c, std::integer_sequence<int, I...>) {
  (comm_tail<NRB - LAG + I>(c), ...);
}

__global__ void __launch_bounds__(512, 2) ag_gemm_kernel(Params p) {
  __shared__ __attribute__((aligned(16))) char smem_raw[LDS_BYTES];
  LAS char* smem = (LAS char*)smem_raw;
  const int b = blockIdx.x, xcd = b & 7, slot = b >> 3;
  const int wave = __builtin_amdgcn_readfirstlane(threadIdx.x >> 6), lane = threadIdx.x & 63;
  const int rank = p.rank & (NR - 1);
  char* my = p.my;
  unsigned* ctr = (unsigned*)(my + CTR_OFF + (size_t)b * 64);
  LAS unsigned* ctl = (LAS unsigned*)(smem + CTL_OFF);
  STAMP(0);
  const unsigned ctr_val = __hip_atomic_load(ctr, __ATOMIC_RELAXED, __HIP_MEMORY_SCOPE_AGENT);
  cfence();
  const __amdgpu_buffer_rsrc_t xr = __builtin_amdgcn_make_buffer_rsrc((void*)p.x, 0, (int)SHARD_BYTES, 0x00020000);

  if (slot < 4) {
    const int q = slot;
    char* pw = p.peer_ws[xcd];
    CommCtx c;
    c.xr = xr;
    c.wave = wave;
    c.self = xcd == 7;
    c.xoff = (lane & 15) * KDIM + (lane >> 4) * 16 + q * 1024;
    u32x4 v[WIN][2];
#pragma unroll
    for (int r = 0; r < INIT; ++r) comm_load(c, r, v[r]);
    cfence();
    const unsigned call = (unsigned)__builtin_amdgcn_readfirstlane(ctr_val) + 1u;
    c.dr = __builtin_amdgcn_make_buffer_rsrc((void*)(pw + STAGE_OFF + (size_t)((call & 1u) * NR + rank) * SHARD_BYTES), 0,
                                             (int)SHARD_BYTES, 0x00020000);
    c.doff = lane * 16 + q * 16384;
    c.flag = (unsigned*)(pw + FLAG_OFF + (size_t)(rank * 4 + q) * 256);
    c.ctr = ctr;
    c.call = call;
    cfence();
    STAMP(1);
    comm_steps(c, v, std::make_integer_sequence<int, NRB>{});
    STAMP(2);
    comm_tails(c, std::make_integer_sequence<int, LAG>{});
    STAMP(3);
    return;
  }

  const int idx = slot - 4;
  const int u_ = (xcd >> 2) * 28 + idx;
  const int panel = (xcd & 3) * 8 + (u_ & 7);
  const int si = u_ >> 3;
  const int src = (rank + 1 + si) & (NR - 1);
  const int nloc = si < 4 ? 5 : 4;
  const int ntot = nloc + NRB;
  const int lc = lane & 15, lg = lane >> 4;
  typedef int i32x2 __attribute__((ext_vector_type(2)));
  typedef unsigned u32x2 __attribute__((ext_vector_type(2)));
  typedef float f32x2 __attribute__((ext_vector_type(2)));

  i32x4 wf[8][4];
  {
    const u32x4 w_srd = make_srd(p.w, W_BYTES);
    const unsigned row = lane >> 2, pc = lane & 3;
    const unsigned sw_e = AGK_WSWZ ? (lg & 1) : 0, sw_o = AGK_WSWZ ? ((lg & 1) | 2) : 0;
    const unsigned vb = (unsigned)(wave * 512 + row) * NSH + panel * 64;
    const unsigned v_even = vb + (pc ^ sw_e) * 16, v_odd = vb + 16 * NSH + (pc ^ sw_o) * 16;
    LAS char* st = smem + wave * WSTAGE;
    const unsigned st_a = (unsigned)(unsigned long long)st;
    if (AGK_WDELAY) __builtin_amdgcn_s_sleep(AGK_WDELAY);
#pragma unroll
    for (int t = 0; t < 4; ++t) dma_w(w_srd, st_a + t * 4096, v_even, v_odd, (unsigned)t * 64u * NSH);
    if (threadIdx.x < 64) ctl[threadIdx.x] = 0u;
    const unsigned q = lc >> 1, rsw = AGK_WSWZ ? (((q >> 2) & 1) | ((lg & 1) << 1)) : 0;
    const LAS char* rb = st + (16 * lg + q) * 64 + 8 * (lane & 1);
    auto take = [&](int t) {
#pragma unroll
      for (int cg = 0; cg < 4; ++cg) {
        const LAS char* a = rb + ((cg ^ rsw) * 16) + (t & 3) * 4096;
        const i32x2 lo = __builtin_amdgcn_ds_read_tr8_b64_v2i32((LAS i32x2*)a);
        const i32x2 hi = __builtin_amdgcn_ds_read_tr8_b64_v2i32((LAS i32x2*)(a + 512));
        wf[t][cg] = (i32x4){lo[0], lo[1], hi[0], hi[1]};
      }
      wait_lgkm0();
      asm volatile("" : "+v"(wf[t][0]), "+v"(wf[t][1]), "+v"(wf[t][2]), "+v"(wf[t][3]));
    };
#pragma unroll
    for (int t = 0; t < 4; ++t) {
      wait_vm<12>();
      take(t);
      dma_w(w_srd, st_a + t * 4096, v_even, v_odd, (unsigned)(t + 4) * 64u * NSH);
    }
    wait_vm<12>(); take(4);
    wait_vm<8>();  take(5);
    wait_vm<4>();  take(6);
    wait_vm<0>();  take(7);
  }
  const float scale = p.sx[0] * p.sw[0];
  unsigned call = (unsigned)__builtin_amdgcn_readfirstlane(ctr_val) + 1u;
  asm volatile("" : "+s"(call));
  STAMP(1);

  const __amdgpu_buffer_rsrc_t sr = __builtin_amdgcn_make_buffer_rsrc(
      (void*)(my + STAGE_OFF + (size_t)(call & 1u) * NR * SHARD_BYTES), 0, (int)(NR * SHARD_BYTES), 0x00020000);
  const __amdgpu_buffer_rsrc_t orr = __builtin_amdgcn_make_buffer_rsrc((void*)p.out, 0, (int)OUT_BYTES, 0x00020000);
  const __amdgpu_buffer_rsrc_t fr = __builtin_amdgcn_make_buffer_rsrc((void*)(my + FLAG_OFF), 0, NR * 4 * 256, 0x00020000);
  const unsigned foff_l = (unsigned)(((lane < 32 ? src : rank) * 4 + (wave >> 1)) * 256 + (lane & 31) * 4);
  const unsigned foff_r = lane < 32 ? foff_l : 0x7fffffffu;
  auto poll = [&](bool with_local) -> unsigned {
    return __builtin_amdgcn_raw_buffer_load_b32(fr, with_local ? foff_l : foff_r, 0, AUX_SYS);
  };
  auto seen = [&](unsigned v) -> unsigned long long { return __ballot(v != 0u && (int)(v - call) >= 0); };
  unsigned* tmo = (unsigned*)(my + TMO_OFF);
  LAS unsigned* dead = ctl;
  LAS unsigned* filled = ctl + 16;
  const unsigned filled_a = (unsigned)(unsigned long long)filled;
  const unsigned voff_s = lane * 16 + wave * 8192;
  const unsigned wr_off = wave * PWAVE + lg * PBLK + lc * 16;
  const unsigned rd_off = ((lane & 31) >> 1) * PBLK + (2 * wave + (lane >> 5)) * 16 + (lane & 1) * 8;
  const unsigned voff_o = (unsigned)((2 * wave + (lane >> 5)) * NSH + panel * 64 + (lane & 31) * 2) * 4u;

  unsigned seq;
  {
    const int d = lane - 7, i = d / 5;
    const bool isl = d >= 0 && d == 5 * i && i < nloc;
    int before = d <= 0 ? 0 : (d + 4) / 5;
    before = before < nloc ? before : nloc;
    const int bit = isl ? 32 + si + 7 * i : lane - before;
    seq = lane < ntot ? (unsigned)(((isl ? rank : src) * NRB + (bit & 31)) << 8 | bit) : (256u << 8);
  }
  unsigned long long ready = 0ull;
  bool lost = false;
  auto wait_bit = [&](int bit) {
    if (ready >> bit & 1ull) return;
    const unsigned long long t0 = __builtin_amdgcn_s_memrealtime();
    bool gave_up = false;
    for (unsigned spins = 1;; ++spins) {
      cfence();
      ready |= seen(poll(bit >= 32));
      if (ready >> bit & 1ull) break;
      if ((spins & 31u) == 0u) {
        const unsigned tv = (unsigned)__builtin_amdgcn_readfirstlane(__hip_atomic_load(tmo, __ATOMIC_RELAXED, __HIP_MEMORY_SCOPE_AGENT));
        if (tv == call || __builtin_amdgcn_s_memrealtime() - t0 > DEADLINE_TICKS) { gave_up = true; break; }
      }
      __builtin_amdgcn_s_sleep(1);
    }
    cfence();
    if (gave_up) {
      __hip_atomic_store(tmo, call, __ATOMIC_RELAXED, __HIP_MEMORY_SCOPE_AGENT);
      __hip_atomic_store(dead, 1u, __ATOMIC_RELAXED, __HIP_MEMORY_SCOPE_WORKGROUP);
      ready = ~0ull;
      lost = true;
      __builtin_amdgcn_s_waitcnt(0x0F70);
    }
  };
  auto load_frag = [&](unsigned grb, int t) -> i32x4 {
    return __builtin_bit_cast(i32x4, __builtin_amdgcn_raw_buffer_load_b128(sr, voff_s + (t & 3) * 1024, grb * RB_BYTES + (t >> 2) * 4096, AGK_XAUX));
  };
  auto entry = [&](int pos) -> unsigned { return (unsigned)__builtin_amdgcn_readlane((int)seq, pos); };

  i32x2 part[8];
  unsigned dv;
  auto reduce_read = [&](int j) {
    const LAS char* pb = smem + (j & (NS - 1)) * PART_BYTES + rd_off;
#pragma unroll
    for (int w = 0; w < 8; ++w) part[w] = *(const LAS i32x2*)(pb + w * PWAVE);
    dv = __hip_atomic_load(dead, __ATOMIC_RELAXED, __HIP_MEMORY_SCOPE_WORKGROUP);
  };
  auto reduce_store = [&](int j) {
    const unsigned grb = entry(j & 63) >> 8;
    const i32x2 s = ((part[0] + part[1]) + (part[2] + part[3])) + ((part[4] + part[5]) + (part[6] + part[7]));
    const float sc = dv ? __builtin_nanf("") : scale;
    f32x2 y;
    y[0] = (float)s[0] * sc;
    y[1] = (float)s[1] * sc;
    __builtin_amdgcn_raw_buffer_store_b64(__builtin_bit_cast(u32x2, y), orr, voff_o + grb * (16u * NSH * 4u), 0, AGK_OAUX);
  };

  i32x4 xa[8], xb[8];
  unsigned pollv;
  {
    const u32x2 z = {0u, 0u};
    const unsigned e0 = entry(0), e1 = entry(1);
    wait_bit(e0 & 63);
    cfence();
#pragma unroll
    for (int t = 0; t < 8; ++t) {
      xa[t] = load_frag(e0 >> 8, t);
      if (t == 6) __builtin_amdgcn_raw_buffer_store_b64(z, orr, OUT_BYTES, 0, 0);
      cfence();
    }
    wait_bit(e1 & 63);
    cfence();
    pollv = poll(false);
    cfence();
#pragma unroll
    for (int t = 0; t < 8; ++t) {
      xb[t] = load_frag(e1 >> 8, t);
      if (t == 6) __builtin_amdgcn_raw_buffer_store_b64(z, orr, OUT_BYTES, 0, 0);
      cfence();
    }
  }
  cfence();
  __builtin_amdgcn_s_barrier();
  cfence();
  if (threadIdx.x == 0) __hip_atomic_store(ctr, call, __ATOMIC_RELAXED, __HIP_MEMORY_SCOPE_AGENT);
  if (lost) __hip_atomic_store(dead, 1u, __ATOMIC_RELAXED, __HIP_MEMORY_SCOPE_WORKGROUP);

  auto step = [&](i32x4 (&xf)[8], int j) {
    i32x4 acc[4];
#pragma unroll
    for (int g = 0; g < 4; ++g) acc[g] = (i32x4){0, 0, 0, 0};
    if (AGK_PRIO == 1) __builtin_amdgcn_s_setprio(1);
    if (AGK_PRIO == 2) __builtin_amdgcn_s_setprio(0);
#pragma unroll
    for (int t = 0; t < 8; ++t)
#pragma unroll
      for (int g = 0; g < 4; ++g) acc[g] = __builtin_amdgcn_mfma_i32_16x16x64_i8(wf[t][g], xf[t], acc[g], 0, 0, 0);
    __builtin_amdgcn_sched_barrier(0);
    ready |= seen(pollv);
    const unsigned e = entry(j + 2);
    if (AGK_PRIO == 1) __builtin_amdgcn_s_setprio(0);
    if (AGK_PRIO == 2) __builtin_amdgcn_s_setprio(1);
    __builtin_amdgcn_sched_barrier(0);
    asm volatile("" ::"v"(xf[0]), "v"(xf[1]), "v"(xf[2]), "v"(xf[3]), "v"(xf[4]), "v"(xf[5]), "v"(xf[6]), "v"(xf[7]), "s"(e), "s"(ready));
    asm volatile("" : "+v"(acc[0]), "+v"(acc[1]), "+v"(acc[2]), "+v"(acc[3]));
    __builtin_amdgcn_s_barrier();
    __builtin_amdgcn_sched_barrier(0);
    cfence();
    wait_bit(e & 63);
    cfence();
    pollv = poll((entry(j + 3) & 32u) != 0u);
    cfence();
    LAS char* pw = smem + (j & 1) * PART_BYTES + wr_off;
    const LAS char* pb = smem + ((j - 1) & 1) * PART_BYTES + rd_off;
#pragma unroll
    for (int t = 0; t < 8; ++t) {
      xf[t] = load_frag(e >> 8, t);
      if (t < 4) {
        *(LAS i32x4*)(pw + t * 4 * PBLK) = acc[t];
        part[2 * t] = *(const LAS i32x2*)(pb + 2 * t * PWAVE);
        part[2 * t + 1] = *(const LAS i32x2*)(pb + (2 * t + 1) * PWAVE);
      }
      if (t == 4) dv = __hip_atomic_load(dead, __ATOMIC_RELAXED, __HIP_MEMORY_SCOPE_WORKGROUP);
      if (t == 6) reduce_store(j - 1);
      __builtin_amdgcn_sched_barrier(0);
    }
    cfence();
    wait_lgkm0();
    __builtin_amdgcn_s_barrier();
    __builtin_amdgcn_sched_barrier(0);
    cfence();
  };

  if (wave >= 4) __builtin_amdgcn_s_barrier();
  for (int j = 0; j + 1 < ntot; j += 2) {
    step(xa, j);
    step(xb, j + 1);
  }
  if (ntot & 1) step(xa, ntot - 1);
  if (wave < 4) __builtin_amdgcn_s_barrier();
  cfence();
  reduce_read(ntot - 1);
  reduce_store(ntot - 1);
  STAMP(47);
}

}

extern "C" void kernel_launch(int rank, int nranks, ncclComm_t comm,
                              hipStream_t stream,
                              void* const* d_in, const int* in_sizes, int n_in,
                              void* d_out, int out_size,
                              const int* in_dtypes, int out_dtype,
                              void* const* d_out_all,
                              void* const* d_ws_all, size_t ws_bytes) {
  (void)comm; (void)in_dtypes; (void)out_dtype; (void)d_out_all;
  if (nranks != agk::NR || (unsigned)rank >= (unsigned)agk::NR || n_in < 4 || ws_bytes < agk::WS_NEED ||
      in_sizes[0] != agk::MSH * agk::KDIM || in_sizes[1] != agk::KDIM * agk::NSH ||
      out_size != agk::NR * agk::MSH * agk::NSH) {
    fprintf(stderr, "kernel_launch: unexpected problem shape\n");
    return;
  }
  agk::Params p;
  p.x = (const char*)d_in[0];
  p.w = (const char*)d_in[1];
  p.sx = (const float*)d_in[2];
  p.sw = (const float*)d_in[3];
  p.out = (float*)d_out;
  p.my = (char*)d_ws_all[rank];
  for (int i = 0; i < agk::NR; ++i) p.peer_ws[i] = (char*)d_ws_all[(rank + 1 + i) % agk::NR];
  p.rank = rank;
  hipLaunchKernelGGL(agk::ag_gemm_kernel, dim3(256), dim3(512), 0, stream, p);
  HIP_CHECK(hipGetLastError());
}
```

```cpp
#include <hip/hip_runtime.h>
#include <rccl/rccl.h>
#include <cstdio>
#include <cstdint>
#include <utility>
#include <type_traits>

#define HIP_CHECK(expr) do { hipError_t _e = (expr); if (_e != hipSuccess) { \
    fprintf(stderr, "HIP error %s at %s:%d\n", hipGetErrorString(_e), __FILE__, __LINE__); \
    return; } } while (0)

namespace agk {

typedef int i32x4 __attribute__((ext_vector_type(4)));
typedef unsigned u32x4 __attribute__((ext_vector_type(4)));
typedef float f32x4 __attribute__((ext_vector_type(4)));
#define LAS __attribute__((address_space(3)))

constexpr int NR = 8;
constexpr int KDIM = 4096, MSH = 512, NSH = 2048;
constexpr int NRB = MSH / 16;
constexpr unsigned RB_BYTES = 16u * KDIM;
constexpr unsigned SHARD_BYTES = (unsigned)MSH * KDIM;
constexpr unsigned W_BYTES = (unsigned)KDIM * NSH;
constexpr unsigned OUT_BYTES = (unsigned)NR * MSH * NSH * 4u;

constexpr size_t CTR_OFF = 0;
constexpr size_t TMO_OFF = 16384;
constexpr size_t FLAG_OFF = 32768;
constexpr size_t STAMP_OFF = 65536;
constexpr size_t STAGE_OFF = 1u << 20;
constexpr size_t WS_NEED = STAGE_OFF + 2ull * NR * SHARD_BYTES;

constexpr unsigned long long DEADLINE_TICKS = 50000000ull;

#define AGK_RLAG 1
constexpr int RLAG = AGK_RLAG;
constexpr int NS = 2 * RLAG;
constexpr int PBLK = 272;
constexpr int PWAVE = 16 * PBLK;
constexpr int PART_BYTES = 8 * PWAVE;
constexpr int WSTAGE = 16384;
constexpr int CTL_OFF = 140 * 1024;
constexpr int LDS_BYTES = CTL_OFF + 256;
static_assert(NS * PART_BYTES <= CTL_OFF && 8 * WSTAGE <= CTL_OFF, "LDS map");

constexpr int AUX_SYS = 17;
#define AGK_XAUX 17
#define AGK_OAUX 0
#define AGK_WIN 8
#define AGK_LAG 1
#define AGK_CD 4
#define AGK_CW 3
#define AGK_CFIRST 1
#define AGK_STAG 1
#define AGK_FLAGOP 0
#define AGK_LSLEEP 32
#define AGK_RSLEEP 1
#define AGK_L0 11
#define AGK_WT 5
#define AGK_GW 1
#define AGK_QROT 0
#define AGK_ROT 1
#define AGK_WAUX 0
#define AGK_SELFAUX 16
#define AGK_ORDER 1
#define AGK_PRIO 0
#define AGK_WSWZ 1
#define AGK_RT 5
#define AGK_MAP 2
#define AGK_WDELAY 0

struct Params {
  const char* x;
  const char* w;
  const float* sx;
  const float* sw;
  float* out;
  char* my;
  char* peer_ws[NR];
  int rank;
};

template <int N>
__device__ __forceinline__ void wait_vm() {
  static_assert(N >= 0 && N < 64, "vmcnt range");
  asm volatile("s_waitcnt vmcnt(%0)" ::"n"(N) : "memory");
}
__device__ __forceinline__ void wait_lgkm0() { asm volatile("s_waitcnt lgkmcnt(0)" ::: "memory"); }
__device__ __forceinline__ void cfence() { asm volatile("" ::: "memory"); }
__device__ __forceinline__ void lds_wait_ge(unsigned a, unsigned need) {
  unsigned tv, ts;
  asm volatile(
      "1:\n\t"
      "ds_read_b32 %0, %2\n\t"
      "s_waitcnt lgkmcnt(0)\n\t"
      "v_readfirstlane_b32 %1, %0\n\t"
      "s_cmp_lt_u32 %1, %3\n\t"
      "s_cbranch_scc1 1b"
      : "=&v"(tv), "=&s"(ts) : "v"(a), "s"(need) : "scc", "memory");
}

__device__ __forceinline__ void lds_inc(unsigned a) {
  asm volatile("s_mov_b64 exec, 1\n\tds_add_u32 %0, %1\n\ts_mov_b64 exec, -1" ::"v"(a), "v"(1u) : "memory");
}
__device__ __forceinline__ u32x4 make_srd(const void* base, unsigned bytes) {
  const unsigned long long a = (unsigned long long)base;
  u32x4 r;
  r.x = (unsigned)__builtin_amdgcn_readfirstlane((unsigned)a);
  r.y = (unsigned)__builtin_amdgcn_readfirstlane((unsigned)(a >> 32) & 0xffffu);
  r.z = bytes;
  r.w = 0x00020000u;
  return r;
}
__device__ __forceinline__ void dma_w(u32x4 srd, unsigned lds, unsigned v_even, unsigned v_odd, unsigned soff) {
  asm volatile(
      "s_mov_b32 m0, %0\n\ts_nop 3\n\t"
      "buffer_load_dwordx4 %1, %3, %4 offen lds\n\t"
      "s_add_u32 m0, m0, 1024\n\ts_nop 0\n\t"
      "buffer_load_dwordx4 %2, %3, %4 offen lds\n\t"
      "s_add_u32 m0, m0, 1024\n\ts_nop 0\n\t"
      "buffer_load_dwordx4 %1, %3, %5 offen lds\n\t"
      "s_add_u32 m0, m0, 1024\n\ts_nop 0\n\t"
      "buffer_load_dwordx4 %2, %3, %5 offen lds"
      ::"s"(lds), "v"(v_even), "v"(v_odd), "s"(srd), "s"(soff), "s"(soff + 32u * NSH)
      : "memory", "scc");
}

#define STAMP(idx) do { } while (0)
#define TS(v) do { } while (0)
#define TSW(v, idx) do { } while (0)

#define SEG(i) do { } while (0)

struct CommCtx {
  u32x4 x_srd;
  __amdgpu_buffer_rsrc_t dr;
  unsigned xoff, doff;
  unsigned ring_a;
  const LAS char* rd0;
  const LAS char* rd1;
  unsigned* flag;
  unsigned call;
};

constexpr int CD = AGK_CD;
constexpr int CRING = 8;
static_assert(CD >= 3 && CD < CRING, "comm ring");

__device__ __forceinline__ void comm_load(const CommCtx& c, int rb) {
  asm volatile(
      "s_mov_b32 m0, %0\n\ts_nop 3\n\t"
      "buffer_load_dwordx4 %1, %2, %3 offen lds\n\t"
      "s_add_u32 m0, m0, 1024\n\ts_nop 0\n\t"
      "buffer_load_dwordx4 %1, %2, %4 offen lds"
      ::"s"(c.ring_a + (rb & (CRING - 1)) * 2048), "v"(c.xoff), "s"(c.x_srd), "s"(rb * RB_BYTES), "s"(rb * RB_BYTES + 8u * KDIM)
      : "memory", "scc");
}
__device__ __forceinline__ void comm_read(const CommCtx& c, int rb, u32x4 (&v)[2]) {
  v[0] = *(const LAS u32x4*)(c.rd0 + (rb & (CRING - 1)) * 2048);
  v[1] = *(const LAS u32x4*)(c.rd1 + (rb & (CRING - 1)) * 2048);
}
__device__ __forceinline__ void comm_store(const CommCtx& c, int rb, int f, u32x4 v) {
  __builtin_amdgcn_raw_buffer_store_b128(v, c.dr, c.doff + f * 1024, rb * RB_BYTES, AUX_SYS);
  cfence();
}
__device__ __forceinline__ void comm_signal(const CommCtx& c, int r) {
  __builtin_amdgcn_s_barrier();
  cfence();
  if ((int)threadIdx.x == (r & 7) * 64) {
    __hip_atomic_store(c.flag + r, c.call, __ATOMIC_RELAXED, __HIP_MEMORY_SCOPE_SYSTEM);
  }
  cfence();
}
__device__ __forceinline__ void comm_step(const CommCtx& c, int r, u32x4 (&cur)[2]) {
  comm_store(c, r, 0, cur[0]);
  wait_vm<4>();
  comm_store(c, r, 1, cur[1]);
  comm_load(c, r + CD);
  wait_vm<6>();
  comm_read(c, r + 1, cur);
  comm_signal(c, r - 1);
}

__global__ void __launch_bounds__(512, 2) ag_gemm_kernel(Params p) {
  __shared__ __attribute__((aligned(16))) char smem_raw[LDS_BYTES];
  LAS char* smem = (LAS char*)smem_raw;
  const int b = blockIdx.x, xcd = b & 7, slot = b >> 3;
  const int wave = __builtin_amdgcn_readfirstlane(threadIdx.x >> 6), lane = threadIdx.x & 63;
  const int rank = p.rank & (NR - 1);
  char* my = p.my;
  unsigned* ctr = (unsigned*)(my + CTR_OFF + (size_t)b * 64);
  LAS unsigned* ctl = (LAS unsigned*)(smem + CTL_OFF);
  TS(ts0);
  const char* const px = p.x;
  const char* const pwt = p.w;
  asm volatile("" ::"s"(px), "s"(pwt), "s"(my), "s"(rank));
  TS(ts1);
  unsigned ctr_val;
  asm volatile("s_load_dword %0, %1, 0x0" : "=s"(ctr_val) : "s"(ctr) : "memory");

  if (__builtin_expect(slot < 4, AGK_CFIRST)) {
    const int q = slot;
    CommCtx c;
    c.x_srd = make_srd(px, SHARD_BYTES);
    const unsigned strip = q * 8 + wave;
    c.xoff = (lane >> 3) * KDIM + strip * 128 + ((lane & 7) ^ (lane >> 3)) * 16;
    LAS char* ring = smem + wave * (CRING * 2048);
    c.ring_a = (unsigned)(unsigned long long)ring;
    comm_load(c, 0);
    comm_load(c, 1);
    TS(ts2);
    c.rd0 = ring + (lane & 15) * 128 + (((lane >> 4)) ^ (lane & 7)) * 16;
    c.rd1 = ring + (lane & 15) * 128 + ((4 + (lane >> 4)) ^ (lane & 7)) * 16;
    char* const pw = p.peer_ws[xcd];
    asm volatile("s_waitcnt lgkmcnt(0)" : "+s"(ctr_val)::"memory");
    const unsigned call = ctr_val + 1u;
    TS(ts3);
    c.dr = __builtin_amdgcn_make_buffer_rsrc((void*)(pw + STAGE_OFF + (size_t)((call & 1u) * NR + rank) * SHARD_BYTES), 0,
                                             (int)SHARD_BYTES, 0x00020000);
    c.doff = lane * 16 + strip * 2048;
    c.flag = (unsigned*)(pw + FLAG_OFF + (size_t)(rank * 4 + q) * 256);
    c.call = call;
    u32x4 v[2];
    wait_vm<2>();
    comm_read(c, 0, v);
    cfence();
    TS(ts4);
#pragma unroll
    for (int r = 2; r < CD; ++r) comm_load(c, r);
    comm_store(c, 0, 0, v[0]);
    comm_store(c, 0, 1, v[1]);
    comm_load(c, CD);
    wait_vm<2 * (CD - 2) + 4>();
    comm_read(c, 1, v);
#pragma nounroll
    for (int r = 1; r < NRB; ++r) comm_step(c, r, v);
    wait_vm<0>();
    comm_signal(c, NRB - 1);
    if (threadIdx.x == 0) __hip_atomic_store(ctr, call, __ATOMIC_RELAXED, __HIP_MEMORY_SCOPE_AGENT);
    STAMP(3);
    TSW(ts0, 0); TSW(ts1, 40); TSW(ts2, 41); TSW(ts3, 42); TSW(ts4, 1);
    return;
  }

  const int idx = slot - 4;
  const int u_ = (xcd >> 2) * 28 + idx;
  const int panel = (xcd & 3) * 8 + (u_ & 7);
  const int si = u_ >> 3;
  const int src = (rank + 1 + si) & (NR - 1);
  const int nloc = si < 4 ? 5 : 4;
  const int ntot = nloc + NRB;
  const int lc = lane & 15, lg = lane >> 4;
  const int rot = AGK_STAG ? (panel + 2 * si) & 7 : 0;
  const int kr = wave ^ ((rot & 1) << 2);
  const int ks0 = rot & 6;
  typedef int i32x2 __attribute__((ext_vector_type(2)));
  typedef unsigned u32x2 __attribute__((ext_vector_type(2)));
  typedef float f32x2 __attribute__((ext_vector_type(2)));

  i32x4 wf[8][4];
  {
    const u32x4 w_srd = make_srd(pwt, W_BYTES);
    const unsigned row = lane >> 2, pc = lane & 3;
    const unsigned sw_e = AGK_WSWZ ? (lg & 1) : 0, sw_o = AGK_WSWZ ? ((lg & 1) | 2) : 0;
    const unsigned vb = (unsigned)(kr * 512 + row) * NSH + panel * 64;
    const unsigned v_even = vb + (pc ^ sw_e) * 16, v_odd = vb + 16 * NSH + (pc ^ sw_o) * 16;
    LAS char* st = smem + wave * WSTAGE;
    const unsigned st_a = (unsigned)(unsigned long long)st;
    if (AGK_WDELAY) __builtin_amdgcn_s_sleep(AGK_WDELAY);
#pragma unroll
    for (int t = 0; t < 4; ++t) dma_w(w_srd, st_a + t * 4096, v_even, v_odd, (unsigned)((t + ks0) & 7) * 64u * NSH);
    if (threadIdx.x < 64) ctl[threadIdx.x] = 0u;
    const unsigned q = lc >> 1, rsw = AGK_WSWZ ? (((q >> 2) & 1) | ((lg & 1) << 1)) : 0;
    const LAS char* rb = st + (16 * lg + q) * 64 + 8 * (lane & 1);
    auto take = [&](int t) {
#pragma unroll
      for (int cg = 0; cg < 4; ++cg) {
        const LAS char* a = rb + ((cg ^ rsw) * 16) + (t & 3) * 4096;
        const i32x2 lo = __builtin_amdgcn_ds_read_tr8_b64_v2i32((LAS i32x2*)a);
        const i32x2 hi = __builtin_amdgcn_ds_read_tr8_b64_v2i32((LAS i32x2*)(a + 512));
        wf[t][cg] = (i32x4){lo[0], lo[1], hi[0], hi[1]};
      }
      wait_lgkm0();
      asm volatile("" : "+v"(wf[t][0]), "+v"(wf[t][1]), "+v"(wf[t][2]), "+v"(wf[t][3]));
    };
#pragma unroll
    for (int t = 0; t < 4; ++t) {
      wait_vm<12>();
      take(t);
      dma_w(w_srd, st_a + t * 4096, v_even, v_odd, (unsigned)((t + 4 + ks0) & 7) * 64u * NSH);
    }
    wait_vm<12>(); take(4);
    wait_vm<8>();  take(5);
    wait_vm<4>();  take(6);
    wait_vm<0>();  take(7);
  }
  const float scale = p.sx[0] * p.sw[0];
  asm volatile("s_waitcnt lgkmcnt(0)" : "+s"(ctr_val)::"memory");
  const unsigned call = ctr_val + 1u;
  STAMP(1);

  const __amdgpu_buffer_rsrc_t sr = __builtin_amdgcn_make_buffer_rsrc(
      (void*)(my + STAGE_OFF + (size_t)(call & 1u) * NR * SHARD_BYTES), 0, (int)(NR * SHARD_BYTES), 0x00020000);
  const __amdgpu_buffer_rsrc_t orr = __builtin_amdgcn_make_buffer_rsrc((void*)p.out, 0, (int)OUT_BYTES, 0x00020000);
  const __amdgpu_buffer_rsrc_t fr = __builtin_amdgcn_make_buffer_rsrc((void*)(my + FLAG_OFF), 0, NR * 4 * 256, 0x00020000);
  const unsigned foff_l = (unsigned)(((lane < 32 ? src : rank) * 4 + (kr >> 1)) * 256 + (lane & 31) * 4);
  const unsigned foff_r = lane < 32 ? foff_l : 0x7fffffffu;
  auto poll = [&](bool with_local) -> unsigned {
    return __builtin_amdgcn_raw_buffer_load_b32(fr, with_local ? foff_l : foff_r, 0, AUX_SYS);
  };
  auto seen = [&](unsigned v) -> unsigned long long { return __ballot(v != 0u && (int)(v - call) >= 0); };
  unsigned* tmo = (unsigned*)(my + TMO_OFF);
  LAS unsigned* dead = ctl;
  LAS unsigned* filled = ctl + 16;
  const unsigned filled_a = (unsigned)(unsigned long long)filled;
  const unsigned voff_s = lane * 16 + kr * 8192;
  const unsigned wr_off = wave * PWAVE + lg * PBLK + lc * 16;
  const unsigned rd_off = ((lane & 31) >> 1) * PBLK + (2 * wave + (lane >> 5)) * 16 + (lane & 1) * 8;
  const unsigned voff_o = (unsigned)((2 * wave + (lane >> 5)) * NSH + panel * 64 + (lane & 31) * 2) * 4u;

  unsigned seq;
  {
    const int d = lane - AGK_L0, i = d / 5;
    const bool isl = d >= 0 && d == 5 * i && i < nloc;
    int before = d <= 0 ? 0 : (d + 4) / 5;
    before = before < nloc ? before : nloc;
    const int bit = isl ? 32 + si + 7 * i : lane - before;
    seq = lane < ntot ? (unsigned)(((isl ? rank : src) * NRB + (bit & 31)) << 8 | bit) : (256u << 8);
  }
  unsigned long long ready = 0ull;
  bool lost = false;
  auto wait_bit = [&](int bit) {
    if (ready >> bit & 1ull) return;
    const unsigned long long t0 = __builtin_amdgcn_s_memrealtime();
    bool gave_up = false;
    for (unsigned spins = 1;; ++spins) {
      cfence();
      ready |= seen(poll(bit >= 32));
      if (ready >> bit & 1ull) break;
      if ((spins & 31u) == 0u) {
        const unsigned tv = (unsigned)__builtin_amdgcn_readfirstlane(__hip_atomic_load(tmo, __ATOMIC_RELAXED, __HIP_MEMORY_SCOPE_AGENT));
        if (tv == call || __builtin_amdgcn_s_memrealtime() - t0 > DEADLINE_TICKS) { gave_up = true; break; }
      }
      if (bit >= 32) __builtin_amdgcn_s_sleep(AGK_LSLEEP); else __builtin_amdgcn_s_sleep(AGK_RSLEEP);
    }
    cfence();
    if (gave_up) {
      __hip_atomic_store(tmo, call, __ATOMIC_RELAXED, __HIP_MEMORY_SCOPE_AGENT);
      __hip_atomic_store(dead, 1u, __ATOMIC_RELAXED, __HIP_MEMORY_SCOPE_WORKGROUP);
      ready = ~0ull;
      lost = true;
      __builtin_amdgcn_s_waitcnt(0x0F70);
    }
  };
  auto load_frag = [&](unsigned grb, int t) -> i32x4 {
    return __builtin_bit_cast(i32x4, __builtin_amdgcn_raw_buffer_load_b128(sr, voff_s + (t & 1) * 1024, grb * RB_BYTES + (((t & 6) + ks0) & 7) * 1024, AGK_XAUX));
  };
  auto entry = [&](int pos) -> unsigned { return (unsigned)__builtin_amdgcn_readlane((int)seq, pos); };

  i32x2 part[8];
  unsigned dv;
  auto reduce_read = [&](int j) {
    const LAS char* pb = smem + (j & (NS - 1)) * PART_BYTES + rd_off;
#pragma unroll
    for (int w = 0; w < 8; ++w) part[w] = *(const LAS i32x2*)(pb + w * PWAVE);
    dv = __hip_atomic_load(dead, __ATOMIC_RELAXED, __HIP_MEMORY_SCOPE_WORKGROUP);
  };
  auto reduce_store = [&](int j) {
    const unsigned grb = entry(j & 63) >> 8;
    const i32x2 s = ((part[0] + part[1]) + (part[2] + part[3])) + ((part[4] + part[5]) + (part[6] + part[7]));
    const float sc = dv ? __builtin_nanf("") : scale;
    f32x2 y;
    y[0] = (float)s[0] * sc;
    y[1] = (float)s[1] * sc;
    __builtin_amdgcn_raw_buffer_store_b64(__builtin_bit_cast(u32x2, y), orr, voff_o + grb * (16u * NSH * 4u), 0, AGK_OAUX);
  };

  i32x4 xa[8], xb[8];
  unsigned pollv;
  {
    const u32x2 z = {0u, 0u};
    const unsigned e0 = entry(0), e1 = entry(1);
    wait_bit(e0 & 63);
    cfence();
#pragma unroll
    for (int t = 0; t < 8; ++t) {
      xa[t] = load_frag(e0 >> 8, t);
      if (t == 6) __builtin_amdgcn_raw_buffer_store_b64(z, orr, OUT_BYTES, 0, 0);
      cfence();
    }
    wait_bit(e1 & 63);
    cfence();
    pollv = poll(false);
    cfence();
#pragma unroll
    for (int t = 0; t < 8; ++t) {
      xb[t] = load_frag(e1 >> 8, t);
      if (t == 6) __builtin_amdgcn_raw_buffer_store_b64(z, orr, OUT_BYTES, 0, 0);
      cfence();
    }
  }
  cfence();
  __builtin_amdgcn_s_barrier();
  cfence();
  if (threadIdx.x == 0) __hip_atomic_store(ctr, call, __ATOMIC_RELAXED, __HIP_MEMORY_SCOPE_AGENT);
  if (lost) __hip_atomic_store(dead, 1u, __ATOMIC_RELAXED, __HIP_MEMORY_SCOPE_WORKGROUP);

  const int rlag = wave < 4 ? 1 : 0;
  auto step = [&](i32x4 (&xf)[8], int j, auto refill) {
    SEG(0);
    LAS char* pwr = smem + (j & 1) * PART_BYTES + wr_off;
    i32x4 acc[4];
#pragma unroll
    for (int h = 0; h < 4 / AGK_GW; ++h) {
#pragma unroll
      for (int g = h * AGK_GW; g < (h + 1) * AGK_GW; ++g) acc[g] = (i32x4){0, 0, 0, 0};
#pragma unroll
      for (int t = 0; t < 8; ++t) {
#pragma unroll
        for (int g = h * AGK_GW; g < (h + 1) * AGK_GW; ++g) acc[g] = __builtin_amdgcn_mfma_i32_16x16x64_i8(wf[t][g], xf[t], acc[g], 0, 0, 0);
        if (t == 7) __builtin_amdgcn_sched_barrier(0);
        if (h > 0 && t >= AGK_WT && t < AGK_WT + AGK_GW) {
          __builtin_amdgcn_sched_barrier(0);
          const int g = (h - 1) * AGK_GW + t - AGK_WT;
          *(LAS i32x4*)(pwr + g * 4 * PBLK) = acc[g];
          __builtin_amdgcn_sched_barrier(0);
        }
      }
    }
    __builtin_amdgcn_sched_barrier(0);
    unsigned e = 0u;
    if constexpr (decltype(refill)::value) {
      ready |= seen(pollv);
      e = entry(j + 2);
    }
#pragma unroll
    for (int g = 4 - AGK_GW; g < 4; ++g) *(LAS i32x4*)(pwr + g * 4 * PBLK) = acc[g];
    wait_lgkm0();
    SEG(1);
    __builtin_amdgcn_s_barrier();
    __builtin_amdgcn_sched_barrier(0);
    cfence();
    SEG(2);
    reduce_read(j - rlag);
    cfence();
    if constexpr (decltype(refill)::value) {
      wait_bit(e & 63);
      cfence();
      pollv = poll((entry(j + 3) & 32u) != 0u);
      cfence();
#pragma unroll
      for (int t = 0; t < 8; ++t) {
        xf[t] = load_frag(e >> 8, t);
        if (t == 4) SEG(3);
        if (t == 6) reduce_store(j - rlag);
        __builtin_amdgcn_sched_barrier(0);
      }
    } else {
      reduce_store(j - rlag);
    }
    cfence();
    SEG(4);
    __builtin_amdgcn_s_barrier();
    __builtin_amdgcn_sched_barrier(0);
    cfence();
  };

  if (wave >= 4) __builtin_amdgcn_s_barrier();
  {
    const std::true_type yes;
    const std::false_type no;
    int j = 0;
    for (; j + 3 < ntot; j += 2) {
      step(xa, j, yes);
      step(xb, j + 1, yes);
    }
    if (ntot & 1) {
      step(xa, j, yes);
      step(xb, j + 1, no);
      step(xa, j + 2, no);
    } else {
      step(xa, j, no);
      step(xb, j + 1, no);
    }
  }
  if (wave < 4) {
    reduce_read(ntot - 1);
    reduce_store(ntot - 1);
  }
  STAMP(47);
  TSW(ts0, 0); TSW(ts1, 40);
}

}

extern "C" void kernel_launch(int rank, int nranks, ncclComm_t comm,
                              hipStream_t stream,
                              void* const* d_in, const int* in_sizes, int n_in,
                              void* d_out, int out_size,
                              const int* in_dtypes, int out_dtype,
                              void* const* d_out_all,
                              void* const* d_ws_all, size_t ws_bytes) {
  (void)comm; (void)in_dtypes; (void)out_dtype; (void)d_out_all;
  if (nranks != agk::NR || (unsigned)rank >= (unsigned)agk::NR || n_in < 4 || ws_bytes < agk::WS_NEED ||
      in_sizes[0] != agk::MSH * agk::KDIM || in_sizes[1] != agk::KDIM * agk::NSH ||
      out_size != agk::NR * agk::MSH * agk::NSH) {
    fprintf(stderr, "kernel_launch: unexpected problem shape\n");
    return;
  }
  agk::Params p;
  p.x = (const char*)d_in[0];
  p.w = (const char*)d_in[1];
  p.sx = (const float*)d_in[2];
  p.sw = (const float*)d_in[3];
  p.out = (float*)d_out;
  p.my = (char*)d_ws_all[rank];
  for (int i = 0; i < agk::NR; ++i) p.peer_ws[i] = (char*)d_ws_all[(rank + 1 + i) % agk::NR];
  p.rank = rank;
  hipLaunchKernelGGL(agk::ag_gemm_kernel, dim3(256), dim3(512), 0, stream, p);
  HIP_CHECK(hipGetLastError());
}
```

```cpp
#include <hip/hip_runtime.h>
#include <rccl/rccl.h>
#include <cstdio>
#include <cstdint>
#include <type_traits>

#define HIP_CHECK(expr) do { hipError_t _e = (expr); if (_e != hipSuccess) { \
    fprintf(stderr, "HIP error %s at %s:%d\n", hipGetErrorString(_e), __FILE__, __LINE__); \
    return; } } while (0)

#define AGK_XAUX 17
#define AGK_OAUX 0
#define AGK_CD 4
#define AGK_CW 2
#define AGK_STAG 1
#define AGK_MAP 2
#define AGK_WDELAY 0
#define AGK_EAGER 1
#define AGK_TOPO 1
#define AGK_WT 5

namespace agk {

typedef int i32x2 __attribute__((ext_vector_type(2)));
typedef int i32x4 __attribute__((ext_vector_type(4)));
typedef unsigned u32x2 __attribute__((ext_vector_type(2)));
typedef unsigned u32x4 __attribute__((ext_vector_type(4)));
typedef float f32x2 __attribute__((ext_vector_type(2)));
#define LAS __attribute__((address_space(3)))

constexpr int NR = 8;
constexpr int KDIM = 4096, MSH = 512, NSH = 2048;
constexpr int NRB = MSH / 16;
constexpr unsigned RB_BYTES = 16u * KDIM;
constexpr unsigned SHARD_BYTES = (unsigned)MSH * KDIM;
constexpr unsigned W_BYTES = (unsigned)KDIM * NSH;
constexpr unsigned OUT_BYTES = (unsigned)NR * MSH * NSH * 4u;
constexpr int NLOC = 5;
constexpr int NTOT = NLOC + NRB;

constexpr size_t CTR_OFF = 0;
constexpr size_t TMO_OFF = 16384;
constexpr size_t FLAG_OFF = 32768;
constexpr size_t STAMP_OFF = 65536;
constexpr size_t STAGE_OFF = 2u << 20;
constexpr size_t WS_NEED = STAGE_OFF + 2ull * NR * SHARD_BYTES;

constexpr unsigned long long DEADLINE_TICKS = 50000000ull;

constexpr int PBLK = 272;
constexpr int PWAVE = 16 * PBLK;
constexpr int PART_BYTES = 8 * PWAVE;
constexpr int STA_OFF = 0;
constexpr int STB_OFF = 72 * 1024;
constexpr int CTL_OFF = STB_OFF + 8 * 8192;
constexpr int LDS_BYTES = CTL_OFF + 256;
static_assert(2 * PART_BYTES <= STB_OFF, "LDS map");

constexpr int AUX_SYS = 17;

constexpr int IDLE_XCD = 1;
constexpr signed char PEER_OF_XCD[NR][8] = {
    {6, -1, 2, 3, 1, 5, 4, 7}, {7, -1, 4, 5, 0, 3, 2, 6}, {3, -1, 0, 6, 4, 5, 1, 7}, {2, -1, 0, 6, 1, 5, 4, 7},
    {5, -1, 1, 7, 0, 2, 3, 6}, {4, -1, 1, 7, 2, 3, 0, 6}, {0, -1, 2, 3, 4, 7, 1, 5}, {1, -1, 4, 5, 3, 6, 0, 2}};
constexpr bool peers_ok() {
  for (int r = 0; r < NR; ++r) {
    unsigned m = 1u << r;
    for (int x = 0; x < 8; ++x) { if (x == IDLE_XCD) continue; if (PEER_OF_XCD[r][x] < 0) return false; m |= 1u << PEER_OF_XCD[r][x]; }
    if (m != 0xffu) return false;
  }
  return true;
}
static_assert(peers_ok(), "every rank pushes to each of its seven peers exactly once");

struct Params {
  const char* x;
  const char* w;
  const float* sx;
  const float* sw;
  float* out;
  char* my;
  char* peer_ws[NR];
  int rank;
};

template <int N>
__device__ __forceinline__ void wait_vm() {
  static_assert(N >= 0 && N < 64, "vmcnt range");
  asm volatile("s_waitcnt vmcnt(%0)" ::"n"(N) : "memory");
}
__device__ __forceinline__ void wait_lgkm0() { asm volatile("s_waitcnt lgkmcnt(0)" ::: "memory"); }
__device__ __forceinline__ void cfence() { asm volatile("" ::: "memory"); }
__device__ __forceinline__ unsigned lds_addr(const LAS void* p) { return (unsigned)(unsigned long long)p; }

__device__ __forceinline__ u32x4 make_srd(const void* base, unsigned bytes) {
  const unsigned long long a = (unsigned long long)base;
  u32x4 r;
  r.x = (unsigned)__builtin_amdgcn_readfirstlane((unsigned)a);
  r.y = (unsigned)__builtin_amdgcn_readfirstlane((unsigned)(a >> 32) & 0xffffu);
  r.z = bytes;
  r.w = 0x00020000u;
  return r;
}
__device__ __forceinline__ void dma2(u32x4 srd, unsigned lds, unsigned voff, unsigned s0, unsigned s1) {
  asm volatile(
      "s_mov_b32 m0, %0\n\ts_nop 3\n\t"
      "buffer_load_dwordx4 %1, %2, %3 offen lds\n\t"
      "s_add_u32 m0, m0, 1024\n\ts_nop 0\n\t"
      "buffer_load_dwordx4 %1, %2, %4 offen lds"
      ::"s"(lds), "v"(voff), "s"(srd), "s"(s0), "s"(s1)
      : "memory", "scc");
}
__device__ __forceinline__ void dma_w(u32x4 srd, unsigned lds, unsigned v_even, unsigned v_odd, unsigned soff) {
  asm volatile(
      "s_mov_b32 m0, %0\n\ts_nop 3\n\t"
      "buffer_load_dwordx4 %1, %3, %4 offen lds\n\t"
      "s_add_u32 m0, m0, 1024\n\ts_nop 0\n\t"
      "buffer_load_dwordx4 %2, %3, %4 offen lds\n\t"
      "s_add_u32 m0, m0, 1024\n\ts_nop 0\n\t"
      "buffer_load_dwordx4 %1, %3, %5 offen lds\n\t"
      "s_add_u32 m0, m0, 1024\n\ts_nop 0\n\t"
      "buffer_load_dwordx4 %2, %3, %5 offen lds"
      ::"s"(lds), "v"(v_even), "v"(v_odd), "s"(srd), "s"(soff), "s"(soff + 32u * NSH)
      : "memory", "scc");
}

#define STAMP(idx) do { } while (0)
#define TS(v) do { } while (0)
#define TSW(v, idx) do { } while (0)

__device__ __forceinline__ unsigned strip_src(int lane) { return (lane >> 3) * KDIM + ((lane & 7) ^ (lane >> 3)) * 16; }
__device__ __forceinline__ unsigned strip_frag(int lane, int f) { return (lane & 15) * 128 + ((4 * f + (lane >> 4)) ^ (lane & 7)) * 16; }

struct CommCtx {
  u32x4 x_srd;
  __amdgpu_buffer_rsrc_t dr;
  unsigned xoff, doff;
  unsigned ring_a;
  const LAS char* rd0;
  const LAS char* rd1;
  unsigned* flag;
  unsigned call;
};

constexpr int CD = AGK_CD;
constexpr int CRING = 8;
static_assert(CD >= 3 && CD < CRING && 8 * CRING * 2048 <= LDS_BYTES, "comm ring");

__device__ __forceinline__ void comm_load(const CommCtx& c, int rb) {
  dma2(c.x_srd, c.ring_a + (rb & (CRING - 1)) * 2048, c.xoff, rb * RB_BYTES, rb * RB_BYTES + 8u * KDIM);
}
__device__ __forceinline__ void comm_read(const CommCtx& c, int rb, u32x4 (&v)[2]) {
  v[0] = *(const LAS u32x4*)(c.rd0 + (rb & (CRING - 1)) * 2048);
  v[1] = *(const LAS u32x4*)(c.rd1 + (rb & (CRING - 1)) * 2048);
}
__device__ __forceinline__ void comm_store(const CommCtx& c, int rb, int f, u32x4 v) {
  __builtin_amdgcn_raw_buffer_store_b128(v, c.dr, c.doff + f * 1024 + rb * RB_BYTES, 0, AUX_SYS);
  cfence();
}
__device__ __forceinline__ void comm_signal(const CommCtx& c, int r) {
  __builtin_amdgcn_s_barrier();
  cfence();
  if ((int)threadIdx.x == (r & 7) * 64) {
    __hip_atomic_store(c.flag + r, c.call, __ATOMIC_RELAXED, __HIP_MEMORY_SCOPE_SYSTEM);
  }
  cfence();
}
__device__ __forceinline__ void comm_step(const CommCtx& c, int r, u32x4 (&cur)[2]) {
  wait_vm<3>();
  comm_store(c, r, 0, cur[0]);
  wait_vm<3>();
  comm_store(c, r, 1, cur[1]);
  comm_load(c, r + CD);
  comm_read(c, r + 1, cur);
  comm_signal(c, r - 1);
}

template <int V> using ic = std::integral_constant<int, V>;
enum { FILL_NONE = 0, FILL_LOCAL = 1, FILL_REMOTE = 2, FILL_TRY = 3, FILL_LATE = 4 };

__global__ void __launch_bounds__(512, 2) ag_gemm_kernel(Params p) {
  __shared__ __attribute__((aligned(16))) char smem_raw[LDS_BYTES];
  LAS char* smem = (LAS char*)smem_raw;
  const int b = blockIdx.x, xcd = b & 7, slot = b >> 3;
  const int wave = __builtin_amdgcn_readfirstlane(threadIdx.x >> 6), lane = threadIdx.x & 63;
  const int rank = p.rank & (NR - 1);
  char* my = p.my;
  unsigned* ctr = (unsigned*)(my + CTR_OFF + (size_t)b * 64);
  TS(ts0);
  const char* const px = p.x;
  const char* const pwt = p.w;
  asm volatile("" ::"s"(px), "s"(pwt), "s"(my), "s"(rank));
  unsigned ctr_val;
  asm volatile("s_load_dword %0, %1, 0x0" : "=s"(ctr_val) : "s"(ctr) : "memory");
  const u32x4 x_srd = make_srd(px, SHARD_BYTES);

  if (__builtin_expect(slot < 4, 1)) {
    const int q = slot;
    if (xcd == IDLE_XCD) return;
    const unsigned strip = q * 8 + wave;
    CommCtx c;
    c.x_srd = x_srd;
    c.xoff = strip_src(lane) + strip * 128;
    LAS char* ring = smem + wave * (CRING * 2048);
    c.ring_a = lds_addr(ring);
    comm_load(c, 0);
    comm_load(c, 1);
    TS(ts2);
    c.rd0 = ring + strip_frag(lane, 0);
    c.rd1 = ring + strip_frag(lane, 1);
    char* const pw = p.peer_ws[xcd];
    asm volatile("s_waitcnt lgkmcnt(0)" : "+s"(ctr_val)::"memory");
    const unsigned call = ctr_val + 1u;
    TS(ts3);
    c.dr = __builtin_amdgcn_make_buffer_rsrc((void*)(pw + STAGE_OFF + (size_t)((call & 1u) * NR + rank) * SHARD_BYTES), 0,
                                             (int)SHARD_BYTES, 0x00020000);
    c.doff = lane * 16 + strip * 2048;
    c.flag = (unsigned*)(pw + FLAG_OFF + (size_t)(rank * 4 + q) * 256);
    c.call = call;
    u32x4 v[2];
    wait_vm<2>();
    comm_read(c, 0, v);
    cfence();
    TS(ts4);
#pragma unroll
    for (int r = 2; r < CD; ++r) comm_load(c, r);
    comm_store(c, 0, 0, v[0]);
    comm_store(c, 0, 1, v[1]);
    comm_load(c, CD);
    wait_vm<2 * (CD - 2) + 4>();
    comm_read(c, 1, v);
#pragma nounroll
    for (int r = 1; r < NRB; ++r) comm_step(c, r, v);
    wait_vm<0>();
    comm_signal(c, NRB - 1);
    if (threadIdx.x == 0) __hip_atomic_store(ctr, call, __ATOMIC_RELAXED, __HIP_MEMORY_SCOPE_AGENT);
    STAMP(3);
    TSW(ts0, 0); TSW(ts2, 41); TSW(ts3, 42); TSW(ts4, 1);
    return;
  }

  const int idx = slot - 4;
  const int u_ = (xcd >> 2) * 28 + idx;
  const int panel = (xcd & 3) * 8 + (u_ & 7);
  const int si = u_ >> 3;
  const int src = (rank + 1 + si) & (NR - 1);
  const int lc = lane & 15, lg = lane >> 4;
  const int rot = AGK_STAG ? (panel + 2 * si) & 7 : 0;
  const int kr = wave ^ ((rot & 1) << 2);
  const int ks0 = rot & 6;
  LAS unsigned* dead = (LAS unsigned*)(smem + CTL_OFF);

  LAS char* const sta = smem + STA_OFF + wave * 8192;
  LAS char* const stb = smem + STB_OFF + wave * 8192;
  const unsigned xl_off = strip_src(lane) + kr * 512;
  auto local_load = [&](int i, int h, LAS char* dst) {
    const unsigned so = (unsigned)(si + 7 * i) * RB_BYTES;
#pragma unroll
    for (int s = 0; s < 2; ++s) {
      const unsigned o = so + (unsigned)((4 * h + 2 * s + ks0) & 7) * 64u;
      dma2(x_srd, lds_addr(dst) + s * 2048, xl_off, o, o + 8u * KDIM);
    }
  };
  const unsigned xf0 = strip_frag(lane, 0), xf1 = strip_frag(lane, 1);
  auto local_take = [&](i32x4 (&xf)[8], int h, const LAS char* s) {
    xf[4 * h + 0] = *(const LAS i32x4*)(s + xf0);
    xf[4 * h + 1] = *(const LAS i32x4*)(s + xf1);
    xf[4 * h + 2] = *(const LAS i32x4*)(s + 2048 + xf0);
    xf[4 * h + 3] = *(const LAS i32x4*)(s + 2048 + xf1);
  };

  i32x4 wf[8][4], xa[8], xb[8];
  float sxv, swv;
  {
    const u32x4 w_srd = make_srd(pwt, W_BYTES);
    const unsigned pc = lane & 3;
    const unsigned vb = (unsigned)(kr * 512 + (lane >> 2)) * NSH + panel * 64;
    const unsigned v_even = vb + (pc ^ (lg & 1)) * 16, v_odd = vb + 16 * NSH + (pc ^ ((lg & 1) | 2)) * 16;
    auto ring = [&](int c) -> LAS char* { return ((c & 2) ? sta : stb) + (c & 1) * 4096; };
    auto w_load = [&](int t) { dma_w(w_srd, lds_addr(ring(t)), v_even, v_odd, (unsigned)((t + ks0) & 7) * 64u * NSH); };
    const unsigned q = lc >> 1, rsw = ((q >> 2) & 1) | ((lg & 1) << 1);
    const unsigned wro = (16 * lg + q) * 64 + 8 * (lane & 1);
    auto w_take = [&](int t) {
#pragma unroll
      for (int cg = 0; cg < 4; ++cg) {
        const LAS char* a = ring(t) + wro + (cg ^ rsw) * 16;
        const i32x2 lo = __builtin_amdgcn_ds_read_tr8_b64_v2i32((LAS i32x2*)a);
        const i32x2 hi = __builtin_amdgcn_ds_read_tr8_b64_v2i32((LAS i32x2*)(a + 512));
        wf[t][cg] = (i32x4){lo[0], lo[1], hi[0], hi[1]};
      }
      wait_lgkm0();
      asm volatile("" : "+v"(wf[t][0]), "+v"(wf[t][1]), "+v"(wf[t][2]), "+v"(wf[t][3]));
    };
    if (AGK_WDELAY) __builtin_amdgcn_s_sleep(AGK_WDELAY);
#pragma unroll
    for (int c = 0; c < 4; ++c) local_load(c >> 1, c & 1, ring(c));
    if (threadIdx.x == 0) *dead = 0u;
    asm volatile("s_load_dword %0, %2, 0x0\n\ts_load_dword %1, %3, 0x0" : "=&s"(sxv), "=&s"(swv) : "s"(p.sx), "s"(p.sw) : "memory");
#pragma unroll
    for (int c = 0; c < 4; ++c) {
      wait_vm<12>();
      i32x4(&xf)[8] = (c >> 1) ? xb : xa;
      local_take(xf, c & 1, ring(c));
      wait_lgkm0();
      const int h = 4 * (c & 1);
      asm volatile("" : "+v"(xf[h]), "+v"(xf[h + 1]), "+v"(xf[h + 2]), "+v"(xf[h + 3]));
      w_load(c);
    }
#pragma unroll
    for (int t = 0; t < 4; ++t) { wait_vm<12>(); w_take(t); w_load(t + 4); }
    wait_vm<12>(); w_take(4); local_load(2, 0, stb);
    wait_vm<12>(); w_take(5); local_load(2, 1, stb + 4096);
    wait_vm<12>(); w_take(6);
    wait_vm<8>();  w_take(7);
  }
  TS(ts1);
  asm volatile("s_waitcnt lgkmcnt(0)" : "+s"(ctr_val), "+s"(sxv), "+s"(swv)::"memory");
  const float scale = sxv * swv;
  const unsigned call = ctr_val + 1u;

  const __amdgpu_buffer_rsrc_t sr = __builtin_amdgcn_make_buffer_rsrc(
      (void*)(my + STAGE_OFF + (size_t)((call & 1u) * NR + src) * SHARD_BYTES), 0, (int)SHARD_BYTES, 0x00020000);
  const __amdgpu_buffer_rsrc_t orr = __builtin_amdgcn_make_buffer_rsrc((void*)p.out, 0, (int)OUT_BYTES, 0x00020000);
  const unsigned* flag = (const unsigned*)(my + FLAG_OFF + (size_t)(src * 4 + (kr >> 1)) * 256) + (lane & 31);
  auto poll = [&]() -> unsigned { return __hip_atomic_load(flag, __ATOMIC_RELAXED, __HIP_MEMORY_SCOPE_SYSTEM); };
  auto seen = [&](unsigned v) -> unsigned { return (unsigned)__ballot((int)(v - call) >= 0); };
  unsigned* tmo = (unsigned*)(my + TMO_OFF);
  const unsigned voff_s = lane * 16 + kr * 8192;
  const unsigned wr_off = wave * PWAVE + lg * PBLK + lc * 16;
  const unsigned rd_off = ((lane & 31) >> 1) * PBLK + (2 * wave + (lane >> 5)) * 16 + (lane & 1) * 8;
  const unsigned voff_o = (unsigned)((2 * wave + (lane >> 5)) * NSH + panel * 64 + (lane & 31) * 2) * 4u;

  unsigned ready = 0u;
  auto wait_rb = [&](int rb) {
    if (ready >> rb & 1u) return;
    const unsigned long long t0 = __builtin_amdgcn_s_memrealtime();
    bool gave_up = false;
    for (unsigned spins = 1;; ++spins) {
      ready |= seen(poll());
      if (ready >> rb & 1u) break;
      if ((spins & 31u) == 0u) {
        const unsigned tv = (unsigned)__builtin_amdgcn_readfirstlane(__hip_atomic_load(tmo, __ATOMIC_RELAXED, __HIP_MEMORY_SCOPE_AGENT));
        if (tv == call || __builtin_amdgcn_s_memrealtime() - t0 > DEADLINE_TICKS) { gave_up = true; break; }
      }
      __builtin_amdgcn_s_sleep(1);
    }
    cfence();
    if (gave_up) {
      __hip_atomic_store(tmo, call, __ATOMIC_RELAXED, __HIP_MEMORY_SCOPE_AGENT);
      __hip_atomic_store(dead, 1u, __ATOMIC_RELAXED, __HIP_MEMORY_SCOPE_WORKGROUP);
      ready = ~0u;
      __builtin_amdgcn_s_waitcnt(0x0070);
    }
  };
  auto load_frag = [&](int rb, int t) -> i32x4 {
    return __builtin_bit_cast(i32x4, __builtin_amdgcn_raw_buffer_load_b128(
        sr, voff_s + (t & 1) * 1024, (unsigned)rb * RB_BYTES + (unsigned)(((t & 6) + ks0) & 7) * 1024u, AGK_XAUX));
  };

  i32x2 part[8];
  unsigned dv;
  auto reduce_read = [&](int j) {
    const LAS char* pb = smem + (j & 1) * PART_BYTES + rd_off;
#pragma unroll
    for (int w = 0; w < 8; ++w) part[w] = *(const LAS i32x2*)(pb + w * PWAVE);
    dv = __hip_atomic_load(dead, __ATOMIC_RELAXED, __HIP_MEMORY_SCOPE_WORKGROUP);
  };
  auto reduce_store = [&](int j) {
    const int lb = si + 7 * j;
    const unsigned grb = j >= NLOC ? (unsigned)(src * NRB + j - NLOC) : ((j >= 0 && lb < NRB) ? (unsigned)(rank * NRB + lb) : 256u);
    const i32x2 s = ((part[0] + part[1]) + (part[2] + part[3])) + ((part[4] + part[5]) + (part[6] + part[7]));
    const float sc = dv ? __builtin_nanf("") : scale;
    f32x2 y;
    y[0] = (float)s[0] * sc;
    y[1] = (float)s[1] * sc;
    __builtin_amdgcn_raw_buffer_store_b64(__builtin_bit_cast(u32x2, y), orr, voff_o + grb * (16u * NSH * 4u), 0, AGK_OAUX);
  };

  cfence();
  __builtin_amdgcn_s_barrier();
  cfence();
  if (threadIdx.x == 0) __hip_atomic_store(ctr, call, __ATOMIC_RELAXED, __HIP_MEMORY_SCOPE_AGENT);

  const int rlag = wave < 4 ? 1 : 0;
  unsigned pollv = call - 1u;
  bool last_loaded = false;
  auto step = [&](i32x4 (&xf)[8], i32x4 (&xo)[8], int j, auto fill) {
    constexpr int FILL = decltype(fill)::value;
    LAS char* pwr = smem + (j & 1) * PART_BYTES + wr_off;
    i32x4 acc[4];
#pragma unroll
    for (int g = 0; g < 4; ++g) {
      acc[g] = (i32x4){0, 0, 0, 0};
#pragma unroll
      for (int t = 0; t < 8; ++t) {
        acc[g] = __builtin_amdgcn_mfma_i32_16x16x64_i8(wf[t][g], xf[t], acc[g], 0, 0, 0);
        if (t == 7) __builtin_amdgcn_sched_barrier(0);
        if (g > 0 && t == AGK_WT) {
          __builtin_amdgcn_sched_barrier(0);
          *(LAS i32x4*)(pwr + (g - 1) * 4 * PBLK) = acc[g - 1];
          __builtin_amdgcn_sched_barrier(0);
        }
      }
    }
    if constexpr (FILL >= FILL_REMOTE) ready |= seen(pollv);
    *(LAS i32x4*)(pwr + 3 * 4 * PBLK) = acc[3];
    wait_lgkm0();
    __builtin_amdgcn_s_barrier();
    __builtin_amdgcn_sched_barrier(0);
    cfence();
    reduce_read(j - rlag);
    cfence();
    if constexpr (FILL == FILL_REMOTE) {
      const int rb = j + 2 - NLOC;
      wait_rb(rb);
      cfence();
      pollv = poll();
      cfence();
#pragma unroll
      for (int t = 0; t < 8; ++t) {
        xf[t] = load_frag(rb, t);
        if (t == 6) reduce_store(j - rlag);
        __builtin_amdgcn_sched_barrier(0);
      }
    } else if constexpr (FILL == FILL_LOCAL) {
      if (j == 0) wait_vm<0>(); else wait_vm<1>();
      local_take(xf, 0, stb);
      local_take(xf, 1, stb + 4096);
      wait_lgkm0();
      asm volatile("" : "+v"(xf[0]), "+v"(xf[1]), "+v"(xf[2]), "+v"(xf[3]), "+v"(xf[4]), "+v"(xf[5]), "+v"(xf[6]), "+v"(xf[7]));
      if (j + 3 < NLOC) {
        local_load(j + 3, 0, stb);
        local_load(j + 3, 1, stb + 4096);
      } else {
        pollv = poll();
      }
      cfence();
      reduce_store(j - rlag);
    } else if constexpr (FILL == FILL_TRY) {
      pollv = poll();
      cfence();
      if (ready >> (NRB - 1)) {
#pragma unroll
        for (int t = 0; t < 8; ++t) xf[t] = load_frag(NRB - 1, t);
        last_loaded = true;
      }
      cfence();
      reduce_store(j - rlag);
    } else if constexpr (FILL == FILL_LATE) {
      reduce_store(j - rlag);
      cfence();
      if (!last_loaded) {
        wait_rb(NRB - 1);
        cfence();
#pragma unroll
        for (int t = 0; t < 8; ++t) xo[t] = load_frag(NRB - 1, t);
      }
    } else {
      reduce_store(j - rlag);
    }
    cfence();
    __builtin_amdgcn_s_barrier();
    __builtin_amdgcn_sched_barrier(0);
    cfence();
  };

  if (wave >= 4) __builtin_amdgcn_s_barrier();
  step(xa, xb, 0, ic<FILL_LOCAL>{});
  step(xb, xa, 1, ic<FILL_LOCAL>{});
  step(xa, xb, 2, ic<FILL_LOCAL>{});
  step(xb, xa, 3, ic<FILL_REMOTE>{});
#pragma nounroll
  for (int j = 4; j < NTOT - 3; j += 2) {
    step(xa, xb, j, ic<FILL_REMOTE>{});
    step(xb, xa, j + 1, ic<FILL_REMOTE>{});
  }
  step(xa, xb, NTOT - 3, ic<FILL_TRY>{});
  step(xb, xa, NTOT - 2, ic<FILL_LATE>{});
  step(xa, xb, NTOT - 1, ic<FILL_NONE>{});
  if (wave < 4) {
    reduce_read(NTOT - 1);
    reduce_store(NTOT - 1);
  }
  STAMP(47);
  TSW(ts0, 0); TSW(ts1, 1);
}

}

extern "C" void kernel_launch(int rank, int nranks, ncclComm_t comm,
                              hipStream_t stream,
                              void* const* d_in, const int* in_sizes, int n_in,
                              void* d_out, int out_size,
                              const int* in_dtypes, int out_dtype,
                              void* const* d_out_all,
                              void* const* d_ws_all, size_t ws_bytes) {
  (void)comm; (void)in_dtypes; (void)out_dtype; (void)d_out_all;
  if (nranks != agk::NR || (unsigned)rank >= (unsigned)agk::NR || n_in < 4 || ws_bytes < agk::WS_NEED ||
      in_sizes[0] != agk::MSH * agk::KDIM || in_sizes[1] != agk::KDIM * agk::NSH ||
      out_size != agk::NR * agk::MSH * agk::NSH) {
    fprintf(stderr, "kernel_launch: unexpected problem shape\n");
    return;
  }
  agk::Params p;
  p.x = (const char*)d_in[0];
  p.w = (const char*)d_in[1];
  p.sx = (const float*)d_in[2];
  p.sw = (const float*)d_in[3];
  p.out = (float*)d_out;
  p.my = (char*)d_ws_all[rank];
  for (int x = 0; x < agk::NR; ++x) {
    const int peer = agk::PEER_OF_XCD[rank][x];
    p.peer_ws[x] = (char*)d_ws_all[peer < 0 ? rank : peer];
  }
  p.rank = rank;
  hipLaunchKernelGGL(agk::ag_gemm_kernel, dim3(256), dim3(512), 0, stream, p);
  HIP_CHECK(hipGetLastError());
}
```

```cpp
#include <hip/hip_runtime.h>
#include <rccl/rccl.h>
#include <cstdio>
#include <cstdint>
#include <type_traits>

#define HIP_CHECK(expr) do { hipError_t _e = (expr); if (_e != hipSuccess) { \
    fprintf(stderr, "HIP error %s at %s:%d\n", hipGetErrorString(_e), __FILE__, __LINE__); \
    return; } } while (0)

#define AGK_XAUX 17
#define AGK_OAUX 2
#define AGK_CD 3
#define AGK_CW 2
#define AGK_STAG 1
#define AGK_MAP 2
#define AGK_WDELAY 0
#define AGK_EAGER 1
#define AGK_TOPO 1
#define AGK_PRIO 0
#define AGK_G0NW 0
#define AGK_PF 0
#define AGK_RIM 0
#define AGK_ST 6
#define AGK_RA 1
#define AGK_RB 2
#define AGK_RM 7
#define AGK_CSTART 0
#define AGK_P8 0
#define AGK_P4 0
#define AGK_WPF 0
#define AGK_WT 6

namespace agk {

typedef int i32x2 __attribute__((ext_vector_type(2)));
typedef int i32x4 __attribute__((ext_vector_type(4)));
typedef unsigned u32x2 __attribute__((ext_vector_type(2)));
typedef unsigned u32x4 __attribute__((ext_vector_type(4)));
typedef float f32x2 __attribute__((ext_vector_type(2)));
#define LAS __attribute__((address_space(3)))

constexpr int NR = 8;
constexpr int KDIM = 4096, MSH = 512, NSH = 2048;
constexpr int NRB = MSH / 16;
constexpr unsigned RB_BYTES = 16u * KDIM;
constexpr unsigned SHARD_BYTES = (unsigned)MSH * KDIM;
constexpr unsigned W_BYTES = (unsigned)KDIM * NSH;
constexpr unsigned OUT_BYTES = (unsigned)NR * MSH * NSH * 4u;
constexpr int NLOC = 5;
constexpr int NTOT = NLOC + NRB;

constexpr size_t CTR_OFF = 0;
constexpr size_t TMO_OFF = 16384;
constexpr size_t FLAG_OFF = 32768;
constexpr size_t STAMP_OFF = 65536;
constexpr size_t STAGE_OFF = 2u << 20;
constexpr size_t WS_NEED = STAGE_OFF + 2ull * NR * SHARD_BYTES;

constexpr unsigned long long DEADLINE_TICKS = 50000000ull;

constexpr int PBLK = 272;
constexpr int PWAVE = 16 * PBLK;
constexpr int PART_BYTES = 8 * PWAVE;
constexpr int STA_OFF = 0;
constexpr int STB_OFF = 72 * 1024;
constexpr int CTL_OFF = STB_OFF + 8 * 8192;
constexpr int LDS_BYTES = CTL_OFF + 512;
static_assert(2 * PART_BYTES <= STB_OFF, "LDS map");

constexpr int AUX_SYS = 17;

constexpr int IDLE_XCD = 1;
constexpr signed char PEER_OF_XCD[NR][8] = {
    {6, -1, 2, 3, 1, 5, 4, 7}, {7, -1, 4, 5, 0, 3, 2, 6}, {3, -1, 0, 6, 4, 5, 1, 7}, {2, -1, 0, 6, 1, 5, 4, 7},
    {5, -1, 1, 7, 0, 2, 3, 6}, {4, -1, 1, 7, 2, 3, 0, 6}, {0, -1, 2, 3, 4, 7, 1, 5}, {1, -1, 4, 5, 3, 6, 0, 2}};
constexpr bool peers_ok() {
  for (int r = 0; r < NR; ++r) {
    unsigned m = 1u << r;
    for (int x = 0; x < 8; ++x) { if (x == IDLE_XCD) continue; if (PEER_OF_XCD[r][x] < 0) return false; m |= 1u << PEER_OF_XCD[r][x]; }
    if (m != 0xffu) return false;
  }
  return true;
}
static_assert(peers_ok(), "every rank pushes to each of its seven peers exactly once");

struct Params {
  const char* x;
  const char* w;
  const float* sx;
  const float* sw;
  float* out;
  char* my;
  char* peer_ws[NR];
  int rank;
};

template <int N>
__device__ __forceinline__ void wait_vm() {
  static_assert(N >= 0 && N < 64, "vmcnt range");
  asm volatile("s_waitcnt vmcnt(%0)" ::"n"(N) : "memory");
}
__device__ __forceinline__ void wait_lgkm0() { asm volatile("s_waitcnt lgkmcnt(0)" ::: "memory"); }
__device__ __forceinline__ void cfence() { asm volatile("" ::: "memory"); }
__device__ __forceinline__ unsigned lds_addr(const LAS void* p) { return (unsigned)(unsigned long long)p; }

__device__ __forceinline__ u32x4 make_srd(const void* base, unsigned bytes) {
  const unsigned long long a = (unsigned long long)base;
  u32x4 r;
  r.x = (unsigned)__builtin_amdgcn_readfirstlane((unsigned)a);
  r.y = (unsigned)__builtin_amdgcn_readfirstlane((unsigned)(a >> 32) & 0xffffu);
  r.z = bytes;
  r.w = 0x00020000u;
  return r;
}
__device__ __forceinline__ void dma2(u32x4 srd, unsigned lds, unsigned voff, unsigned s0, unsigned s1) {
  asm volatile(
      "s_mov_b32 m0, %0\n\ts_nop 3\n\t"
      "buffer_load_dwordx4 %1, %2, %3 offen lds\n\t"
      "s_add_u32 m0, m0, 1024\n\ts_nop 0\n\t"
      "buffer_load_dwordx4 %1, %2, %4 offen lds"
      ::"s"(lds), "v"(voff), "s"(srd), "s"(s0), "s"(s1)
      : "memory", "scc");
}
__device__ __forceinline__ void dma_w(u32x4 srd, unsigned lds, unsigned v_even, unsigned v_odd, unsigned soff) {
  asm volatile(
      "s_mov_b32 m0, %0\n\ts_nop 3\n\t"
      "buffer_load_dwordx4 %1, %3, %4 offen lds\n\t"
      "s_add_u32 m0, m0, 1024\n\ts_nop 0\n\t"
      "buffer_load_dwordx4 %2, %3, %4 offen lds\n\t"
      "s_add_u32 m0, m0, 1024\n\ts_nop 0\n\t"
      "buffer_load_dwordx4 %1, %3, %5 offen lds\n\t"
      "s_add_u32 m0, m0, 1024\n\ts_nop 0\n\t"
      "buffer_load_dwordx4 %2, %3, %5 offen lds"
      ::"s"(lds), "v"(v_even), "v"(v_odd), "s"(srd), "s"(soff), "s"(soff + 32u * NSH)
      : "memory", "scc");
}

#define STAMP(idx) do { } while (0)
#define TS(v) do { } while (0)
#define TSW(v, idx) do { } while (0)

#define SEG(i) do { } while (0)

__device__ __forceinline__ unsigned strip_src(int lane) { return (lane >> 3) * KDIM + ((lane & 7) ^ (lane >> 3)) * 16; }
__device__ __forceinline__ unsigned strip_frag(int lane, int f) { return (lane & 15) * 128 + ((4 * f + (lane >> 4)) ^ (lane & 7)) * 16; }

struct CommCtx {
  u32x4 x_srd;
  __amdgpu_buffer_rsrc_t dr;
  unsigned xoff, doff;
  unsigned ring_a;
  const LAS char* rd0;
  const LAS char* rd1;
  unsigned* flag;
  unsigned call;
};

constexpr int CD = AGK_CD;
constexpr int CRING = 8;
static_assert(CD >= 3 && CD < CRING && 8 * CRING * 2048 <= LDS_BYTES, "comm ring");

__device__ __forceinline__ void comm_load(const CommCtx& c, int rb) {
  dma2(c.x_srd, c.ring_a + (rb & (CRING - 1)) * 2048, c.xoff, rb * RB_BYTES, rb * RB_BYTES + 8u * KDIM);
}
__device__ __forceinline__ void comm_read(const CommCtx& c, int rb, u32x4 (&v)[2]) {
  v[0] = *(const LAS u32x4*)(c.rd0 + (rb & (CRING - 1)) * 2048);
  v[1] = *(const LAS u32x4*)(c.rd1 + (rb & (CRING - 1)) * 2048);
}
__device__ __forceinline__ void comm_store(const CommCtx& c, int rb, int f, u32x4 v) {
  __builtin_amdgcn_raw_buffer_store_b128(v, c.dr, c.doff + f * 1024 + rb * RB_BYTES, 0, AUX_SYS);
  cfence();
}
__device__ __forceinline__ void comm_signal(const CommCtx& c, int r) {
  __builtin_amdgcn_s_barrier();
  cfence();
  if ((int)threadIdx.x == (r & 7) * 64) {
    __hip_atomic_store(c.flag + r, c.call, __ATOMIC_RELAXED, __HIP_MEMORY_SCOPE_SYSTEM);
  }
  cfence();
}
__device__ __forceinline__ void comm_step(const CommCtx& c, int r, u32x4 (&cur)[2]) {
  wait_vm<3>();
  comm_store(c, r, 0, cur[0]);
  wait_vm<3>();
  comm_store(c, r, 1, cur[1]);
  comm_load(c, r + CD);
  comm_read(c, r + 1, cur);
  comm_signal(c, r - 1);
}

template <int V> using ic = std::integral_constant<int, V>;
enum { FILL_NONE = 0, FILL_LOCAL = 1, FILL_REMOTE = 2, FILL_TRY = 3, FILL_LATE = 4 };

__global__ void __launch_bounds__(512, 2) ag_gemm_kernel(Params p) {
  __shared__ __attribute__((aligned(16))) char smem_raw[LDS_BYTES];
  LAS char* smem = (LAS char*)smem_raw;
  const int b = blockIdx.x, xcd = b & 7, slot = b >> 3;
  const int wave = __builtin_amdgcn_readfirstlane(threadIdx.x >> 6), lane = threadIdx.x & 63;
  const int rank = p.rank & (NR - 1);
  char* my = p.my;
  unsigned* ctr = (unsigned*)(my + CTR_OFF + (size_t)b * 64);
  TS(ts0);
  const char* const px = p.x;
  const char* const pwt = p.w;
  asm volatile("" ::"s"(px), "s"(pwt), "s"(my), "s"(rank));
  auto next_call = [&]() -> unsigned {
    unsigned v;
    asm volatile("s_load_dword %0, %1, 0x0\n\ts_waitcnt lgkmcnt(0)" : "=s"(v) : "s"(ctr) : "memory");
    return v + 1u;
  };
  const u32x4 x_srd = make_srd(px, SHARD_BYTES);

  if (__builtin_expect(slot < 4, 1)) {
    const int q = slot;
    if (xcd == IDLE_XCD) return;
    const unsigned strip = q * 8 + wave;
    CommCtx c;
    c.x_srd = x_srd;
    c.xoff = strip_src(lane) + strip * 128;
    LAS char* ring = smem + wave * (CRING * 2048);
    c.ring_a = lds_addr(ring);
    comm_load(c, 0);
    comm_load(c, 1);
    TS(ts2);
    c.rd0 = ring + strip_frag(lane, 0);
    c.rd1 = ring + strip_frag(lane, 1);
    const unsigned call = next_call();
    char* const pw = p.peer_ws[xcd];
    TS(ts3);
    c.dr = __builtin_amdgcn_make_buffer_rsrc((void*)(pw + STAGE_OFF + (size_t)((call & 1u) * NR + rank) * SHARD_BYTES), 0,
                                             (int)SHARD_BYTES, 0x00020000);
    c.doff = lane * 16 + strip * 2048;
    c.flag = (unsigned*)(pw + FLAG_OFF + (size_t)(rank * 4 + q) * 256);
    c.call = call;
    u32x4 v[2];
    wait_vm<2>();
    comm_read(c, 0, v);
    cfence();
    TS(ts4);
#pragma unroll
    for (int r = 2; r < CD; ++r) comm_load(c, r);
    comm_store(c, 0, 0, v[0]);
    comm_store(c, 0, 1, v[1]);
    comm_load(c, CD);
    wait_vm<2 * (CD - 2) + 4>();
    comm_read(c, 1, v);
#pragma nounroll
    for (int r = 1; r < NRB; ++r) comm_step(c, r, v);
    wait_vm<0>();
    comm_signal(c, NRB - 1);
    if (threadIdx.x == 0) __hip_atomic_store(ctr, call, __ATOMIC_RELAXED, __HIP_MEMORY_SCOPE_AGENT);
    STAMP(3);
    TSW(ts0, 0); TSW(ts2, 41); TSW(ts3, 42); TSW(ts4, 1);
    return;
  }

  const int idx = slot - 4;
  const int u_ = (xcd >> 2) * 28 + idx;
  const int panel = (xcd & 3) * 8 + (u_ & 7);
  const int si = u_ >> 3;
  const int src = (rank + 1 + si) & (NR - 1);
  const int lc = lane & 15, lg = lane >> 4;
  const int rot = AGK_STAG ? (AGK_RA * panel + AGK_RB * si) & AGK_RM : 0;
  const int kr = wave ^ ((rot & 1) << 2);
  const int ks0 = rot & 6;
  LAS unsigned* dead = (LAS unsigned*)(smem + CTL_OFF);

  LAS char* const sta = smem + STA_OFF + wave * 8192;
  LAS char* const stb = smem + STB_OFF + wave * 8192;
  const unsigned xl_off = strip_src(lane) + kr * 512;
  auto local_load = [&](int i, int h, LAS char* dst) {
    const unsigned so = (unsigned)(si + 7 * i) * RB_BYTES;
#pragma unroll
    for (int s = 0; s < 2; ++s) {
      const unsigned o = so + (unsigned)((4 * h + 2 * s + ks0) & 7) * 64u;
      dma2(x_srd, lds_addr(dst) + s * 2048, xl_off, o, o + 8u * KDIM);
    }
  };
  const unsigned xf0 = strip_frag(lane, 0), xf1 = strip_frag(lane, 1);
  auto local_take = [&](i32x4 (&xf)[8], int h, const LAS char* s) {
    xf[4 * h + 0] = *(const LAS i32x4*)(s + xf0);
    xf[4 * h + 1] = *(const LAS i32x4*)(s + xf1);
    xf[4 * h + 2] = *(const LAS i32x4*)(s + 2048 + xf0);
    xf[4 * h + 3] = *(const LAS i32x4*)(s + 2048 + xf1);
  };

  i32x4 wf[8][4], xa[8], xb[8];
  float sxv, swv;
  unsigned ctr_val;
  {
    const u32x4 w_srd = make_srd(pwt, W_BYTES);
    const unsigned pc = lane & 3;
    const unsigned vb = (unsigned)(kr * 512 + (lane >> 2)) * NSH + panel * 64;
    const unsigned v_even = vb + (pc ^ (lg & 1)) * 16, v_odd = vb + 16 * NSH + (pc ^ ((lg & 1) | 2)) * 16;
    auto ring = [&](int c) -> LAS char* { return ((c & 2) ? sta : stb) + (c & 1) * 4096; };
    auto w_load = [&](int t) { dma_w(w_srd, lds_addr(ring(t)), v_even, v_odd, (unsigned)((t + ks0) & 7) * 64u * NSH); };
    const unsigned q = lc >> 1, rsw = ((q >> 2) & 1) | ((lg & 1) << 1);
    const unsigned wro = (16 * lg + q) * 64 + 8 * (lane & 1);
    auto w_take = [&](int t) {
#pragma unroll
      for (int cg = 0; cg < 4; ++cg) {
        const LAS char* a = ring(t) + wro + (cg ^ rsw) * 16;
        const i32x2 lo = __builtin_amdgcn_ds_read_tr8_b64_v2i32((LAS i32x2*)a);
        const i32x2 hi = __builtin_amdgcn_ds_read_tr8_b64_v2i32((LAS i32x2*)(a + 512));
        wf[t][cg] = (i32x4){lo[0], lo[1], hi[0], hi[1]};
      }
      wait_lgkm0();
      asm volatile("" : "+v"(wf[t][0]), "+v"(wf[t][1]), "+v"(wf[t][2]), "+v"(wf[t][3]));
    };
    if (AGK_WDELAY) __builtin_amdgcn_s_sleep(AGK_WDELAY);
#pragma unroll
    for (int c = 0; c < 4; ++c) local_load(c >> 1, c & 1, ring(c));
    if (threadIdx.x == 0) *dead = 0u;
    asm volatile("s_load_dword %0, %3, 0x0\n\ts_load_dword %1, %4, 0x0\n\ts_load_dword %2, %5, 0x0\n\ts_waitcnt lgkmcnt(0)"
                 : "=&s"(sxv), "=&s"(swv), "=&s"(ctr_val) : "s"(p.sx), "s"(p.sw), "s"(ctr) : "memory");
#pragma unroll
    for (int c = 0; c < 4; ++c) {
      wait_vm<12>();
      i32x4(&xf)[8] = (c >> 1) ? xb : xa;
      local_take(xf, c & 1, ring(c));
      wait_lgkm0();
      const int h = 4 * (c & 1);
      asm volatile("" : "+v"(xf[h]), "+v"(xf[h + 1]), "+v"(xf[h + 2]), "+v"(xf[h + 3]));
      w_load(c);
    }
#pragma unroll
    for (int t = 0; t < 4; ++t) { wait_vm<12>(); w_take(t); w_load(t + 4); }
    wait_vm<12>(); w_take(4); local_load(2, 0, stb);
    wait_vm<12>(); w_take(5); local_load(2, 1, stb + 4096);
    wait_vm<12>(); w_take(6);
    wait_vm<8>();  w_take(7);
  }
  TS(ts1);
  const float scale = sxv * swv;
  const unsigned call = ctr_val + 1u;

  const __amdgpu_buffer_rsrc_t sr = __builtin_amdgcn_make_buffer_rsrc(
      (void*)(my + STAGE_OFF + (size_t)((call & 1u) * NR + src) * SHARD_BYTES), 0, (int)SHARD_BYTES, 0x00020000);
  const __amdgpu_buffer_rsrc_t orr = __builtin_amdgcn_make_buffer_rsrc((void*)p.out, 0, (int)OUT_BYTES, 0x00020000);
  const unsigned* flag = (const unsigned*)(my + FLAG_OFF + (size_t)(src * 4 + (kr >> 1)) * 256) + (lane & 31);
  auto poll = [&]() -> unsigned { return __hip_atomic_load(flag, __ATOMIC_RELAXED, __HIP_MEMORY_SCOPE_SYSTEM); };
  auto seen = [&](unsigned v) -> unsigned { return (unsigned)__ballot((int)(v - call) >= 0); };
  const __amdgpu_buffer_rsrc_t wsr = __builtin_amdgcn_make_buffer_rsrc((void*)my, 0, (int)WS_NEED, 0x00020000);
  const unsigned pf_base = lane < 32 ? (unsigned)(FLAG_OFF + (src * 4 + (kr >> 1)) * 256 + lane * 4)
                                     : (lane < 40 ? (unsigned)STAGE_OFF + ((call & 1u) * NR + src) * SHARD_BYTES + kr * 8192 + (panel & 7) * 1024 + (lane - 32) * 128
                                                  : 0x40000000u);
  const unsigned pf_mask = lane < 32 ? 0u : ~0u;
  auto poll_pf = [&](unsigned off) -> unsigned {
    return __builtin_amdgcn_raw_buffer_load_b32(wsr, pf_base + (pf_mask & off), 0, AUX_SYS);
  };
  unsigned* tmo = (unsigned*)(my + TMO_OFF);
  const unsigned voff_s = lane * 16 + kr * 8192;
  const unsigned wr_off = wave * PWAVE + lg * PBLK + lc * 16;
  const unsigned rd_off = ((lane & 31) >> 1) * PBLK + (2 * wave + (lane >> 5)) * 16 + (lane & 1) * 8;
  const unsigned voff_o = (unsigned)((2 * wave + (lane >> 5)) * NSH + panel * 64 + (lane & 31) * 2) * 4u;

  unsigned ready = 0u;
  const int sh = si == 3 ? 2 * (panel & 3) : (panel & 7);
  auto order = [&](int pos) -> int {
    const int m = pos < AGK_P8 ? 7 : (pos < AGK_P4 ? 3 : 0);
    return (pos & ~m) | ((pos + (m == 3 && si == 3 ? sh >> 1 : sh)) & m);
  };
  auto wait_rb = [&](int rb) {
    if (ready >> rb & 1u) return;
    const unsigned long long t0 = __builtin_amdgcn_s_memrealtime();
    bool gave_up = false;
    for (unsigned spins = 1;; ++spins) {
      ready |= seen(poll());
      if (ready >> rb & 1u) break;
      if ((spins & 31u) == 0u) {
        const unsigned tv = (unsigned)__builtin_amdgcn_readfirstlane(__hip_atomic_load(tmo, __ATOMIC_RELAXED, __HIP_MEMORY_SCOPE_AGENT));
        if (tv == call || __builtin_amdgcn_s_memrealtime() - t0 > DEADLINE_TICKS) { gave_up = true; break; }
      }
      __builtin_amdgcn_s_sleep(1);
    }
    cfence();
    if (gave_up) {
      __hip_atomic_store(tmo, call, __ATOMIC_RELAXED, __HIP_MEMORY_SCOPE_AGENT);
      __hip_atomic_store(dead, 1u, __ATOMIC_RELAXED, __HIP_MEMORY_SCOPE_WORKGROUP);
      ready = ~0u;
      __builtin_amdgcn_s_waitcnt(0x0070);
    }
  };
  auto load_frag = [&](int rb, int t) -> i32x4 {
    return __builtin_bit_cast(i32x4, __builtin_amdgcn_raw_buffer_load_b128(
        sr, voff_s + (t & 1) * 1024, (unsigned)rb * RB_BYTES + (unsigned)(((t & 6) + ks0) & 7) * 1024u, AGK_XAUX));
  };

  i32x2 part[8];
  unsigned dv;
  auto reduce_read = [&](int j) {
    const LAS char* pb = smem + (j & 1) * PART_BYTES + rd_off;
#pragma unroll
    for (int w = 0; w < 8; ++w) part[w] = *(const LAS i32x2*)(pb + w * PWAVE);
    dv = __hip_atomic_load(dead, __ATOMIC_RELAXED, __HIP_MEMORY_SCOPE_WORKGROUP);
  };
  u32x2 yv;
  auto reduce_sum = [&]() {
    const i32x2 s = ((part[0] + part[1]) + (part[2] + part[3])) + ((part[4] + part[5]) + (part[6] + part[7]));
    const float sc = dv ? __builtin_nanf("") : scale;
    f32x2 y;
    y[0] = (float)s[0] * sc;
    y[1] = (float)s[1] * sc;
    yv = __builtin_bit_cast(u32x2, y);
  };
  auto put = [&](int j) {
    const int lb = si + 7 * j;
    const unsigned grb = j >= NLOC ? (unsigned)(src * NRB + order(j - NLOC)) : ((j >= 0 && lb < NRB) ? (unsigned)(rank * NRB + lb) : 256u);
    __builtin_amdgcn_raw_buffer_store_b64(yv, orr, voff_o + grb * (16u * NSH * 4u), 0, AGK_OAUX);
  };
  auto reduce_store = [&](int j) { reduce_sum(); put(j); };

  cfence();
  __builtin_amdgcn_s_barrier();
  cfence();
  if (threadIdx.x == 0) __hip_atomic_store(ctr, call, __ATOMIC_RELAXED, __HIP_MEMORY_SCOPE_AGENT);

  const int rlag = (AGK_RIM || wave < 4) ? 1 : 0;
  unsigned pollv = call - 1u;
  bool last_loaded = false;
  auto step = [&](i32x4 (&xf)[8], i32x4 (&xo)[8], int j, auto fill) {
    constexpr int FILL = decltype(fill)::value;
    SEG(0);
    if (AGK_PRIO == 1) __builtin_amdgcn_s_setprio(1);
    if (AGK_PRIO == 2) __builtin_amdgcn_s_setprio(0);
    LAS char* pwr = smem + (j & 1) * PART_BYTES + wr_off;
    i32x4 acc[4];
#pragma unroll
    for (int g = 0; g < 4; ++g) {
      acc[g] = (i32x4){0, 0, 0, 0};
#pragma unroll
      for (int t = 0; t < 8; ++t) {
        acc[g] = __builtin_amdgcn_mfma_i32_16x16x64_i8(wf[t][g], xf[t], acc[g], 0, 0, 0);
        if (t == 7) __builtin_amdgcn_sched_barrier(0);
        if (t == 7 && g == 0) SEG(1);
        if (g > 0 && t == AGK_WT) {
          __builtin_amdgcn_sched_barrier(0);
          *(LAS i32x4*)(pwr + (g - 1) * 4 * PBLK) = acc[g - 1];
          __builtin_amdgcn_sched_barrier(0);
        }
      }
    }
    if constexpr (FILL >= FILL_REMOTE) ready |= seen(pollv);
    *(LAS i32x4*)(pwr + 3 * 4 * PBLK) = acc[3];
    if (AGK_PRIO == 1) __builtin_amdgcn_s_setprio(0);
    if (AGK_PRIO == 2) __builtin_amdgcn_s_setprio(1);
    wait_lgkm0();
    SEG(2);
    __builtin_amdgcn_s_barrier();
    __builtin_amdgcn_sched_barrier(0);
    cfence();
    SEG(3);
    reduce_read(j - rlag);
    cfence();
    if constexpr (FILL == FILL_REMOTE) {
      const int rb = order(j + 2 - NLOC);
      wait_rb(rb);
      cfence();
      pollv = poll();
      cfence();
#pragma unroll
      for (int t = 0; t < 8; ++t) {
        xf[t] = load_frag(rb, t);
        if (t == AGK_ST) reduce_store(j - rlag);
        __builtin_amdgcn_sched_barrier(0);
      }
    } else if constexpr (FILL == FILL_LOCAL) {
      if (j == 0) wait_vm<0>(); else wait_vm<1>();
      local_take(xf, 0, stb);
      local_take(xf, 1, stb + 4096);
      wait_lgkm0();
      asm volatile("" : "+v"(xf[0]), "+v"(xf[1]), "+v"(xf[2]), "+v"(xf[3]), "+v"(xf[4]), "+v"(xf[5]), "+v"(xf[6]), "+v"(xf[7]));
      if (j + 3 < NLOC) {
        local_load(j + 3, 0, stb);
        local_load(j + 3, 1, stb + 4096);
      } else {
        pollv = poll();
      }
      cfence();
      reduce_store(j - rlag);
    } else if constexpr (FILL == FILL_TRY) {
      pollv = poll();
      cfence();
      if (ready >> (NRB - 1)) {
#pragma unroll
        for (int t = 0; t < 8; ++t) xf[t] = load_frag(NRB - 1, t);
        last_loaded = true;
      }
      cfence();
      reduce_store(j - rlag);
    } else if constexpr (FILL == FILL_LATE) {
      reduce_store(j - rlag);
      cfence();
      if (!last_loaded) {
        wait_rb(NRB - 1);
        cfence();
#pragma unroll
        for (int t = 0; t < 8; ++t) xo[t] = load_frag(NRB - 1, t);
      }
    } else {
      reduce_store(j - rlag);
    }
    cfence();
    SEG(4);
    __builtin_amdgcn_s_barrier();
    __builtin_amdgcn_sched_barrier(0);
    cfence();
  };

  if (wave >= 4) __builtin_amdgcn_s_barrier();
  step(xa, xb, 0, ic<FILL_LOCAL>{});
  step(xb, xa, 1, ic<FILL_LOCAL>{});
  step(xa, xb, 2, ic<FILL_LOCAL>{});
  step(xb, xa, 3, ic<FILL_REMOTE>{});
#pragma nounroll
  for (int j = 4; j < NTOT - 3; j += 2) {
    step(xa, xb, j, ic<FILL_REMOTE>{});
    step(xb, xa, j + 1, ic<FILL_REMOTE>{});
  }
  step(xa, xb, NTOT - 3, ic<FILL_TRY>{});
  step(xb, xa, NTOT - 2, ic<FILL_LATE>{});
  step(xa, xb, NTOT - 1, ic<FILL_NONE>{});
  if (AGK_RIM || wave < 4) {
    reduce_read(NTOT - 1);
    reduce_sum();
    put(NTOT - 1);
  }
  STAMP(47);
  TSW(ts0, 0); TSW(ts1, 1);
}

}

extern "C" void kernel_launch(int rank, int nranks, ncclComm_t comm,
                              hipStream_t stream,
                              void* const* d_in, const int* in_sizes, int n_in,
                              void* d_out, int out_size,
                              const int* in_dtypes, int out_dtype,
                              void* const* d_out_all,
                              void* const* d_ws_all, size_t ws_bytes) {
  (void)comm; (void)in_dtypes; (void)out_dtype; (void)d_out_all;
  if (nranks != agk::NR || (unsigned)rank >= (unsigned)agk::NR || n_in < 4 || ws_bytes < agk::WS_NEED ||
      in_sizes[0] != agk::MSH * agk::KDIM || in_sizes[1] != agk::KDIM * agk::NSH ||
      out_size != agk::NR * agk::MSH * agk::NSH) {
    fprintf(stderr, "kernel_launch: unexpected problem shape\n");
    return;
  }
  agk::Params p;
  p.x = (const char*)d_in[0];
  p.w = (const char*)d_in[1];
  p.sx = (const float*)d_in[2];
  p.sw = (const float*)d_in[3];
  p.out = (float*)d_out;
  p.my = (char*)d_ws_all[rank];
  for (int x = 0; x < agk::NR; ++x) {
    const int peer = agk::PEER_OF_XCD[rank][x];
    p.peer_ws[x] = (char*)d_ws_all[peer < 0 ? rank : peer];
  }
  p.rank = rank;
  hipLaunchKernelGGL(agk::ag_gemm_kernel, dim3(256), dim3(512), 0, stream, p);
  HIP_CHECK(hipGetLastError());
}
```

```cpp
#include <hip/hip_runtime.h>
#include <rccl/rccl.h>
#include <cstdio>
#include <cstdint>
#include <type_traits>

#define HIP_CHECK(expr) do { hipError_t _e = (expr); if (_e != hipSuccess) { \
    fprintf(stderr, "HIP error %s at %s:%d\n", hipGetErrorString(_e), __FILE__, __LINE__); \
    return; } } while (0)

namespace agk {

typedef int i32x2 __attribute__((ext_vector_type(2)));
typedef int i32x4 __attribute__((ext_vector_type(4)));
typedef unsigned u32x2 __attribute__((ext_vector_type(2)));
typedef unsigned u32x4 __attribute__((ext_vector_type(4)));
typedef float f32x2 __attribute__((ext_vector_type(2)));
#define LAS __attribute__((address_space(3)))

constexpr int NR = 8;
constexpr int KDIM = 4096, MSH = 512, NSH = 2048;
constexpr int NRB = MSH / 16;
constexpr unsigned RB_BYTES = 16u * KDIM;
constexpr unsigned SHARD_BYTES = (unsigned)MSH * KDIM;
constexpr unsigned W_BYTES = (unsigned)KDIM * NSH;
constexpr unsigned OUT_BYTES = (unsigned)NR * MSH * NSH * 4u;
constexpr int NLOC = 5;
constexpr int NTOT = NLOC + NRB;

constexpr size_t CTR_OFF = 0;
constexpr size_t TMO_OFF = 16384;
constexpr size_t FLAG_OFF = 32768;
constexpr size_t STAGE_OFF = 2u << 20;
constexpr size_t WS_NEED = STAGE_OFF + 2ull * NR * SHARD_BYTES;

constexpr unsigned long long DEADLINE_TICKS = 50000000ull;

constexpr int PBLK = 272;
constexpr int PWAVE = 16 * PBLK;
constexpr int PART_BYTES = 8 * PWAVE;
constexpr int STA_OFF = 0;
constexpr int STB_OFF = 72 * 1024;
constexpr int CTL_OFF = STB_OFF + 8 * 8192;
constexpr int LDS_BYTES = CTL_OFF + 256;
static_assert(2 * PART_BYTES <= STB_OFF, "LDS map");

constexpr int AUX_SYS = 17;
constexpr int AUX_NT = 2;

constexpr int IDLE_XCD = 1;
constexpr signed char PEER_OF_XCD[NR][8] = {
    {6, -1, 2, 3, 1, 5, 4, 7}, {7, -1, 4, 5, 0, 3, 2, 6}, {3, -1, 0, 6, 4, 5, 1, 7}, {2, -1, 0, 6, 1, 5, 4, 7},
    {5, -1, 1, 7, 0, 2, 3, 6}, {4, -1, 1, 7, 2, 3, 0, 6}, {0, -1, 2, 3, 4, 7, 1, 5}, {1, -1, 4, 5, 3, 6, 0, 2}};
constexpr bool peers_ok() {
  for (int r = 0; r < NR; ++r) {
    unsigned m = 1u << r;
    for (int x = 0; x < 8; ++x) { if (x == IDLE_XCD) continue; if (PEER_OF_XCD[r][x] < 0) return false; m |= 1u << PEER_OF_XCD[r][x]; }
    if (m != 0xffu) return false;
  }
  return true;
}
static_assert(peers_ok(), "every rank pushes to each of its seven peers exactly once");

struct Params {
  const char* x;
  const char* w;
  const float* sx;
  const float* sw;
  float* out;
  char* my;
  char* peer_ws[NR];
  int rank;
};

template <int N>
__device__ __forceinline__ void wait_vm() {
  static_assert(N >= 0 && N < 64, "vmcnt range");
  asm volatile("s_waitcnt vmcnt(%0)" ::"n"(N) : "memory");
}
__device__ __forceinline__ void wait_lgkm0() { asm volatile("s_waitcnt lgkmcnt(0)" ::: "memory"); }
__device__ __forceinline__ void cfence() { asm volatile("" ::: "memory"); }
__device__ __forceinline__ unsigned lds_addr(const LAS void* p) { return (unsigned)(unsigned long long)p; }

__device__ __forceinline__ u32x4 make_srd(const void* base, unsigned bytes) {
  const unsigned long long a = (unsigned long long)base;
  u32x4 r;
  r.x = (unsigned)__builtin_amdgcn_readfirstlane((unsigned)a);
  r.y = (unsigned)__builtin_amdgcn_readfirstlane((unsigned)(a >> 32) & 0xffffu);
  r.z = bytes;
  r.w = 0x00020000u;
  return r;
}
__device__ __forceinline__ void dma2(u32x4 srd, unsigned lds, unsigned voff, unsigned s0, unsigned s1) {
  asm volatile(
      "s_mov_b32 m0, %0\n\ts_nop 3\n\t"
      "buffer_load_dwordx4 %1, %2, %3 offen lds\n\t"
      "s_add_u32 m0, m0, 1024\n\ts_nop 0\n\t"
      "buffer_load_dwordx4 %1, %2, %4 offen lds"
      ::"s"(lds), "v"(voff), "s"(srd), "s"(s0), "s"(s1)
      : "memory", "scc");
}
__device__ __forceinline__ void dma_w(u32x4 srd, unsigned lds, unsigned v_even, unsigned v_odd, unsigned soff) {
  asm volatile(
      "s_mov_b32 m0, %0\n\ts_nop 3\n\t"
      "buffer_load_dwordx4 %1, %3, %4 offen lds\n\t"
      "s_add_u32 m0, m0, 1024\n\ts_nop 0\n\t"
      "buffer_load_dwordx4 %2, %3, %4 offen lds\n\t"
      "s_add_u32 m0, m0, 1024\n\ts_nop 0\n\t"
      "buffer_load_dwordx4 %1, %3, %5 offen lds\n\t"
      "s_add_u32 m0, m0, 1024\n\ts_nop 0\n\t"
      "buffer_load_dwordx4 %2, %3, %5 offen lds"
      ::"s"(lds), "v"(v_even), "v"(v_odd), "s"(srd), "s"(soff), "s"(soff + 32u * NSH)
      : "memory", "scc");
}

__device__ __forceinline__ unsigned strip_src(int lane) { return (lane >> 3) * KDIM + ((lane & 7) ^ (lane >> 3)) * 16; }
__device__ __forceinline__ unsigned strip_frag(int lane, int f) { return (lane & 15) * 128 + ((4 * f + (lane >> 4)) ^ (lane & 7)) * 16; }

struct CommCtx {
  u32x4 x_srd;
  __amdgpu_buffer_rsrc_t dr;
  unsigned xoff, doff;
  unsigned ring_a;
  const LAS char* rd0;
  const LAS char* rd1;
  unsigned* flag;
  unsigned call;
};

constexpr int CD = 3;
constexpr int CRING = 8;
static_assert(CD >= 3 && CD < CRING && 8 * CRING * 2048 <= LDS_BYTES, "comm ring");

__device__ __forceinline__ void comm_load(const CommCtx& c, int rb) {
  dma2(c.x_srd, c.ring_a + (rb & (CRING - 1)) * 2048, c.xoff, rb * RB_BYTES, rb * RB_BYTES + 8u * KDIM);
}
__device__ __forceinline__ void comm_read(const CommCtx& c, int rb, u32x4 (&v)[2]) {
  v[0] = *(const LAS u32x4*)(c.rd0 + (rb & (CRING - 1)) * 2048);
  v[1] = *(const LAS u32x4*)(c.rd1 + (rb & (CRING - 1)) * 2048);
}
__device__ __forceinline__ void comm_store(const CommCtx& c, int rb, int f, u32x4 v) {
  __builtin_amdgcn_raw_buffer_store_b128(v, c.dr, c.doff + f * 1024 + rb * RB_BYTES, 0, AUX_SYS);
  cfence();
}
__device__ __forceinline__ void comm_signal(const CommCtx& c, int r) {
  __builtin_amdgcn_s_barrier();
  cfence();
  if ((int)threadIdx.x == (r & 7) * 64) __hip_atomic_store(c.flag + r, c.call, __ATOMIC_RELAXED, __HIP_MEMORY_SCOPE_SYSTEM);
  cfence();
}
__device__ __forceinline__ void comm_step(const CommCtx& c, int r, u32x4 (&cur)[2]) {
  wait_vm<3>();
  comm_store(c, r, 0, cur[0]);
  wait_vm<3>();
  comm_store(c, r, 1, cur[1]);
  comm_load(c, r + CD);
  comm_read(c, r + 1, cur);
  comm_signal(c, r - 1);
}

template <int V> using ic = std::integral_constant<int, V>;
enum { FILL_NONE = 0, FILL_LOCAL = 1, FILL_REMOTE = 2 };

__global__ void __launch_bounds__(512, 2) ag_gemm_kernel(Params p) {
  __shared__ __attribute__((aligned(16))) char smem_raw[LDS_BYTES];
  LAS char* smem = (LAS char*)smem_raw;
  const int b = blockIdx.x, xcd = b & 7, slot = b >> 3;
  const int wave = __builtin_amdgcn_readfirstlane(threadIdx.x >> 6), lane = threadIdx.x & 63;
  const int rank = p.rank & (NR - 1);
  char* my = p.my;
  unsigned* ctr = (unsigned*)(my + CTR_OFF + (size_t)b * 64);
  const char* const px = p.x;
  const char* const pwt = p.w;
  asm volatile("" ::"s"(px), "s"(pwt), "s"(my), "s"(rank));
  const u32x4 x_srd = make_srd(px, SHARD_BYTES);

  if (__builtin_expect(slot < 4, 1)) {
    const int q = slot;
    if (xcd == IDLE_XCD) return;
    const unsigned strip = q * 8 + wave;
    CommCtx c;
    c.x_srd = x_srd;
    c.xoff = strip_src(lane) + strip * 128;
    LAS char* ring = smem + wave * (CRING * 2048);
    c.ring_a = lds_addr(ring);
    comm_load(c, 0);
    comm_load(c, 1);
    c.rd0 = ring + strip_frag(lane, 0);
    c.rd1 = ring + strip_frag(lane, 1);
    unsigned call;
    asm volatile("s_load_dword %0, %1, 0x0\n\ts_waitcnt lgkmcnt(0)" : "=s"(call) : "s"(ctr) : "memory");
    call += 1u;
    char* const pw = p.peer_ws[xcd];
    c.dr = __builtin_amdgcn_make_buffer_rsrc((void*)(pw + STAGE_OFF + (size_t)((call & 1u) * NR + rank) * SHARD_BYTES), 0,
                                             (int)SHARD_BYTES, 0x00020000);
    c.doff = lane * 16 + strip * 2048;
    c.flag = (unsigned*)(pw + FLAG_OFF + (size_t)(rank * 4 + q) * 256);
    c.call = call;
    u32x4 v[2];
    wait_vm<2>();
    comm_read(c, 0, v);
    cfence();
#pragma unroll
    for (int r = 2; r < CD; ++r) comm_load(c, r);
    comm_store(c, 0, 0, v[0]);
    comm_store(c, 0, 1, v[1]);
    comm_load(c, CD);
    wait_vm<2 * (CD - 2) + 4>();
    comm_read(c, 1, v);
#pragma nounroll
    for (int r = 1; r < NRB; ++r) comm_step(c, r, v);
    wait_vm<0>();
    comm_signal(c, NRB - 1);
    if (threadIdx.x == 0) __hip_atomic_store(ctr, call, __ATOMIC_RELAXED, __HIP_MEMORY_SCOPE_AGENT);
    return;
  }

  const int u_ = (xcd >> 2) * 28 + slot - 4;
  const int panel = (xcd & 3) * 8 + (u_ & 7);
  const int si = u_ >> 3;
  const int src = (rank + 1 + si) & (NR - 1);
  const int lc = lane & 15, lg = lane >> 4;
  const int rot = (panel + 2 * si) & 7;
  const int kr = wave ^ ((rot & 1) << 2);
  const int ks0 = rot & 6;
  LAS unsigned* dead = (LAS unsigned*)(smem + CTL_OFF);

  LAS char* const sta = smem + STA_OFF + wave * 8192;
  LAS char* const stb = smem + STB_OFF + wave * 8192;
  const unsigned xl_off = strip_src(lane) + kr * 512;
  auto local_load = [&](int i, int h, LAS char* dst) {
    const unsigned so = (unsigned)(si + 7 * i) * RB_BYTES;
#pragma unroll
    for (int s = 0; s < 2; ++s) {
      const unsigned o = so + (unsigned)((4 * h + 2 * s + ks0) & 7) * 64u;
      dma2(x_srd, lds_addr(dst) + s * 2048, xl_off, o, o + 8u * KDIM);
    }
  };
  const unsigned xf0 = strip_frag(lane, 0), xf1 = strip_frag(lane, 1);
  auto local_take = [&](i32x4 (&xf)[8], int h, const LAS char* s) {
    xf[4 * h + 0] = *(const LAS i32x4*)(s + xf0);
    xf[4 * h + 1] = *(const LAS i32x4*)(s + xf1);
    xf[4 * h + 2] = *(const LAS i32x4*)(s + 2048 + xf0);
    xf[4 * h + 3] = *(const LAS i32x4*)(s + 2048 + xf1);
  };

  i32x4 wf[8][4], xa[8], xb[8], la[2][4];
  float sxv, swv;
  unsigned ctr_val;
  {
    const u32x4 w_srd = make_srd(pwt, W_BYTES);
    const unsigned pc = lane & 3;
    const unsigned vb = (unsigned)(kr * 512 + (lane >> 2)) * NSH + panel * 64;
    const unsigned v_even = vb + (pc ^ (lg & 1)) * 16, v_odd = vb + 16 * NSH + (pc ^ ((lg & 1) | 2)) * 16;
    auto ring = [&](int c) -> LAS char* { return ((c & 2) ? sta : stb) + (c & 1) * 4096; };
    auto w_load = [&](int t) { dma_w(w_srd, lds_addr(ring(t)), v_even, v_odd, (unsigned)((t + ks0) & 7) * 64u * NSH); };
    const unsigned q = lc >> 1, rsw = ((q >> 2) & 1) | ((lg & 1) << 1);
    const unsigned wro = (16 * lg + q) * 64 + 8 * (lane & 1);
    auto w_take = [&](int t) {
#pragma unroll
      for (int cg = 0; cg < 4; ++cg) {
        const LAS char* a = ring(t) + wro + (cg ^ rsw) * 16;
        const i32x2 lo = __builtin_amdgcn_ds_read_tr8_b64_v2i32((LAS i32x2*)a);
        const i32x2 hi = __builtin_amdgcn_ds_read_tr8_b64_v2i32((LAS i32x2*)(a + 512));
        wf[t][cg] = (i32x4){lo[0], lo[1], hi[0], hi[1]};
      }
      wait_lgkm0();
      asm volatile("" : "+v"(wf[t][0]), "+v"(wf[t][1]), "+v"(wf[t][2]), "+v"(wf[t][3]));
    };
#pragma unroll
    for (int c = 0; c < 4; ++c) local_load(c >> 1, c & 1, ring(c));
    if (threadIdx.x == 0) *dead = 0u;
    asm volatile("s_load_dword %0, %3, 0x0\n\ts_load_dword %1, %4, 0x0\n\ts_load_dword %2, %5, 0x0\n\ts_waitcnt lgkmcnt(0)"
                 : "=&s"(sxv), "=&s"(swv), "=&s"(ctr_val) : "s"(p.sx), "s"(p.sw), "s"(ctr) : "memory");
#pragma unroll
    for (int c = 0; c < 4; ++c) {
      wait_vm<12>();
      i32x4(&xf)[8] = (c >> 1) ? xb : xa;
      local_take(xf, c & 1, ring(c));
      wait_lgkm0();
      const int h = 4 * (c & 1);
      asm volatile("" : "+v"(xf[h]), "+v"(xf[h + 1]), "+v"(xf[h + 2]), "+v"(xf[h + 3]));
      w_load(c);
    }
    auto mul01 = [&](int t) {
#pragma unroll
      for (int g = 0; g < 4; ++g) {
        if (t == 0) {
          asm volatile("v_mfma_i32_16x16x64_i8 %0, %1, %2, 0" : "=&v"(la[0][g]) : "v"(wf[t][g]), "v"(xa[t]));
          asm volatile("v_mfma_i32_16x16x64_i8 %0, %1, %2, 0" : "=&v"(la[1][g]) : "v"(wf[t][g]), "v"(xb[t]));
        } else {
          asm volatile("v_mfma_i32_16x16x64_i8 %0, %1, %2, %0" : "+v"(la[0][g]) : "v"(wf[t][g]), "v"(xa[t]));
          asm volatile("v_mfma_i32_16x16x64_i8 %0, %1, %2, %0" : "+v"(la[1][g]) : "v"(wf[t][g]), "v"(xb[t]));
        }
      }
    };
#pragma unroll
    for (int t = 0; t < 4; ++t) { wait_vm<12>(); w_take(t); w_load(t + 4); mul01(t); }
    wait_vm<12>(); w_take(4); local_load(2, 0, stb); mul01(4);
    wait_vm<12>(); w_take(5); local_load(2, 1, stb + 4096); mul01(5);
    wait_vm<12>(); w_take(6); mul01(6);
    wait_vm<8>();  w_take(7); mul01(7);
    asm volatile("s_nop 7\n\ts_nop 7" ::: "memory");
  }
  const float scale = sxv * swv;
  const unsigned call = ctr_val + 1u;

  const __amdgpu_buffer_rsrc_t sr = __builtin_amdgcn_make_buffer_rsrc(
      (void*)(my + STAGE_OFF + (size_t)((call & 1u) * NR + src) * SHARD_BYTES), 0, (int)SHARD_BYTES, 0x00020000);
  const __amdgpu_buffer_rsrc_t orr = __builtin_amdgcn_make_buffer_rsrc((void*)p.out, 0, (int)OUT_BYTES, 0x00020000);
  const unsigned* flag = (const unsigned*)(my + FLAG_OFF + (size_t)(src * 4 + (kr >> 1)) * 256) + (lane & 31);
  auto poll = [&]() -> unsigned { return __hip_atomic_load(flag, __ATOMIC_RELAXED, __HIP_MEMORY_SCOPE_SYSTEM); };
  auto seen = [&](unsigned v) -> unsigned { return (unsigned)__ballot((int)(v - call) >= 0); };
  unsigned* tmo = (unsigned*)(my + TMO_OFF);
  const unsigned voff_s = lane * 16 + kr * 8192;
  const unsigned wr_off = wave * PWAVE + lg * PBLK + lc * 16;
  const unsigned rd_off = ((lane & 31) >> 1) * PBLK + (2 * wave + (lane >> 5)) * 16 + (lane & 1) * 8;
  const unsigned voff_o = (unsigned)((2 * wave + (lane >> 5)) * NSH + panel * 64 + (lane & 31) * 2) * 4u;

  unsigned ready = 0u;
  auto wait_rb = [&](int rb) {
    if (ready >> rb & 1u) return;
    const unsigned long long t0 = __builtin_amdgcn_s_memrealtime();
    bool gave_up = false;
    for (unsigned spins = 1;; ++spins) {
      ready |= seen(poll());
      if (ready >> rb & 1u) break;
      if ((spins & 31u) == 0u) {
        const unsigned tv = (unsigned)__builtin_amdgcn_readfirstlane(__hip_atomic_load(tmo, __ATOMIC_RELAXED, __HIP_MEMORY_SCOPE_AGENT));
        if (tv == call || __builtin_amdgcn_s_memrealtime() - t0 > DEADLINE_TICKS) { gave_up = true; break; }
      }
      __builtin_amdgcn_s_sleep(1);
    }
    cfence();
    if (gave_up) {
      __hip_atomic_store(tmo, call, __ATOMIC_RELAXED, __HIP_MEMORY_SCOPE_AGENT);
      __hip_atomic_store(dead, 1u, __ATOMIC_RELAXED, __HIP_MEMORY_SCOPE_WORKGROUP);
      ready = ~0u;
      __builtin_amdgcn_s_waitcnt(0x0070);
    }
  };
  auto load_frag = [&](int rb, int t) -> i32x4 {
    return __builtin_bit_cast(i32x4, __builtin_amdgcn_raw_buffer_load_b128(
        sr, voff_s + (t & 1) * 1024, (unsigned)rb * RB_BYTES + (unsigned)(((t & 6) + ks0) & 7) * 1024u, AUX_SYS));
  };

  i32x2 part[8];
  unsigned dv;
  auto reduce_read = [&](int j) {
    const LAS char* pb = smem + (j & 1) * PART_BYTES + rd_off;
#pragma unroll
    for (int w = 0; w < 8; ++w) part[w] = *(const LAS i32x2*)(pb + w * PWAVE);
    dv = __hip_atomic_load(dead, __ATOMIC_RELAXED, __HIP_MEMORY_SCOPE_WORKGROUP);
  };
  auto reduce_store = [&](int j) {
    const i32x2 s = ((part[0] + part[1]) + (part[2] + part[3])) + ((part[4] + part[5]) + (part[6] + part[7]));
    const float sc = dv ? __builtin_nanf("") : scale;
    f32x2 y;
    y[0] = (float)s[0] * sc;
    y[1] = (float)s[1] * sc;
    const int lb = si + 7 * j;
    const unsigned grb = j >= NLOC ? (unsigned)(src * NRB + j - NLOC) : ((j >= 0 && lb < NRB) ? (unsigned)(rank * NRB + lb) : 256u);
    __builtin_amdgcn_raw_buffer_store_b64(__builtin_bit_cast(u32x2, y), orr, voff_o + grb * (16u * NSH * 4u), 0, AUX_NT);
  };

  cfence();
  __builtin_amdgcn_s_barrier();
  cfence();
  if (threadIdx.x == 0) __hip_atomic_store(ctr, call, __ATOMIC_RELAXED, __HIP_MEMORY_SCOPE_AGENT);
#pragma unroll
  for (int i = 0; i < 2; ++i)
#pragma unroll
    for (int g = 0; g < 4; ++g) *(LAS i32x4*)(smem + i * PART_BYTES + wr_off + g * 4 * PBLK) = la[i][g];
  wait_vm<0>();
  local_take(xa, 0, stb);
  local_take(xa, 1, stb + 4096);
  wait_lgkm0();
  asm volatile("" : "+v"(xa[0]), "+v"(xa[1]), "+v"(xa[2]), "+v"(xa[3]), "+v"(xa[4]), "+v"(xa[5]), "+v"(xa[6]), "+v"(xa[7]));
  local_load(3, 0, stb);
  local_load(3, 1, stb + 4096);
  __builtin_amdgcn_s_barrier();
  cfence();
  reduce_read(0);
  reduce_store(0);
  cfence();
  reduce_read(1);
  reduce_store(1);
  cfence();
  __builtin_amdgcn_s_barrier();
  cfence();

  const int rlag = wave < 4 ? 1 : 0;
  unsigned pollv = call - 1u;
  auto step = [&](i32x4 (&xf)[8], i32x4 (&xo)[8], int j, auto fill) {
    constexpr int FILL = decltype(fill)::value;
    LAS char* pwr = smem + (j & 1) * PART_BYTES + wr_off;
    i32x4 acc[4];
#pragma unroll
    for (int g = 0; g < 4; ++g) {
      acc[g] = (i32x4){0, 0, 0, 0};
#pragma unroll
      for (int t = 0; t < 8; ++t) {
        acc[g] = __builtin_amdgcn_mfma_i32_16x16x64_i8(wf[t][g], xf[t], acc[g], 0, 0, 0);
        if (t == 7) __builtin_amdgcn_sched_barrier(0);
        if (g > 0 && t == 6) {
          __builtin_amdgcn_sched_barrier(0);
          *(LAS i32x4*)(pwr + (g - 1) * 4 * PBLK) = acc[g - 1];
          __builtin_amdgcn_sched_barrier(0);
        }
      }
    }
    if constexpr (FILL == FILL_REMOTE) ready |= seen(pollv);
    *(LAS i32x4*)(pwr + 3 * 4 * PBLK) = acc[3];
    if (wave >= 4) wait_lgkm0();
    cfence();
    __builtin_amdgcn_s_barrier();
    __builtin_amdgcn_sched_barrier(0);
    cfence();
    reduce_read(j - rlag);
    cfence();
    if constexpr (FILL == FILL_REMOTE) {
      if (j == 3) {
        wait_vm<2>();
        local_take(xo, 0, stb);
        local_take(xo, 1, stb + 4096);
      }
      const int rb = j + 2 - NLOC;
      if (rb < NRB) wait_rb(rb);
      cfence();
      pollv = poll();
      cfence();
#pragma unroll
      for (int t = 0; t < 8; ++t) {
        xf[t] = load_frag(rb, t);
        if (t == 6) reduce_store(j - rlag);
        __builtin_amdgcn_sched_barrier(0);
      }
    } else if constexpr (FILL == FILL_LOCAL) {
      wait_vm<2>();
      local_take(xo, 0, stb);
      local_take(xo, 1, stb + 4096);
      wait_lgkm0();
      asm volatile("" : "+v"(xo[0]), "+v"(xo[1]), "+v"(xo[2]), "+v"(xo[3]), "+v"(xo[4]), "+v"(xo[5]), "+v"(xo[6]), "+v"(xo[7]));
      local_load(4, 0, stb);
      local_load(4, 1, stb + 4096);
      pollv = poll();
      cfence();
      reduce_store(rlag ? -1 : j);
    } else {
      reduce_store(j - rlag);
      if (wave >= 4) return;
    }
    wait_lgkm0();
    __builtin_amdgcn_s_barrier();
    __builtin_amdgcn_sched_barrier(0);
    cfence();
  };

  if (wave >= 4) __builtin_amdgcn_s_barrier();
  step(xa, xb, 2, ic<FILL_LOCAL>{});
  step(xb, xa, 3, ic<FILL_REMOTE>{});
#pragma nounroll
  for (int j = 4; j < NTOT - 1; j += 2) {
    step(xa, xb, j, ic<FILL_REMOTE>{});
    step(xb, xa, j + 1, ic<FILL_REMOTE>{});
  }
  step(xa, xb, NTOT - 1, ic<FILL_NONE>{});
  if (wave < 4) {
    reduce_read(NTOT - 1);
    reduce_store(NTOT - 1);
  }
}

}

extern "C" void kernel_launch(int rank, int nranks, ncclComm_t comm,
                              hipStream_t stream,
                              void* const* d_in, const int* in_sizes, int n_in,
                              void* d_out, int out_size,
                              const int* in_dtypes, int out_dtype,
                              void* const* d_out_all,
                              void* const* d_ws_all, size_t ws_bytes) {
  (void)comm; (void)in_dtypes; (void)out_dtype; (void)d_out_all;
  if (nranks != agk::NR || (unsigned)rank >= (unsigned)agk::NR || n_in < 4 || ws_bytes < agk::WS_NEED ||
      in_sizes[0] != agk::MSH * agk::KDIM || in_sizes[1] != agk::KDIM * agk::NSH ||
      out_size != agk::NR * agk::MSH * agk::NSH) {
    fprintf(stderr, "kernel_launch: unexpected problem shape\n");
    return;
  }
  agk::Params p;
  p.x = (const char*)d_in[0];
  p.w = (const char*)d_in[1];
  p.sx = (const float*)d_in[2];
  p.sw = (const float*)d_in[3];
  p.out = (float*)d_out;
  p.my = (char*)d_ws_all[rank];
  for (int x = 0; x < agk::NR; ++x) {
    const int peer = agk::PEER_OF_XCD[rank][x];
    p.peer_ws[x] = (char*)d_ws_all[peer < 0 ? rank : peer];
  }
  p.rank = rank;
  hipLaunchKernelGGL(agk::ag_gemm_kernel, dim3(256), dim3(512), 0, stream, p);
  HIP_CHECK(hipGetLastError());
}
```

```cpp
#include <hip/hip_runtime.h>
#include <rccl/rccl.h>
#include <cstdio>
#include <cstdint>
#include <type_traits>

#define HIP_CHECK(expr) do { hipError_t _e = (expr); if (_e != hipSuccess) { \
    fprintf(stderr, "HIP error %s at %s:%d\n", hipGetErrorString(_e), __FILE__, __LINE__); \
    return; } } while (0)

namespace agk {

typedef int i32x2 __attribute__((ext_vector_type(2)));
typedef int i32x4 __attribute__((ext_vector_type(4)));
typedef unsigned u32x2 __attribute__((ext_vector_type(2)));
typedef unsigned u32x4 __attribute__((ext_vector_type(4)));
typedef float f32x2 __attribute__((ext_vector_type(2)));
#define LAS __attribute__((address_space(3)))

constexpr int NR = 8;
constexpr int KDIM = 4096, MSH = 512, NSH = 2048;
constexpr int NRB = MSH / 16;
constexpr unsigned RB_BYTES = 16u * KDIM;
constexpr unsigned SHARD_BYTES = (unsigned)MSH * KDIM;
constexpr unsigned W_BYTES = (unsigned)KDIM * NSH;
constexpr unsigned OUT_BYTES = (unsigned)NR * MSH * NSH * 4u;
constexpr int NLOC = 5;
constexpr int NTOT = NLOC + NRB;

constexpr size_t CTR_OFF = 0;
constexpr size_t TMO_OFF = 16384;
constexpr size_t FLAG_OFF = 32768;
constexpr size_t STAGE_OFF = 2u << 20;
constexpr size_t WS_NEED = STAGE_OFF + 2ull * NR * SHARD_BYTES;

constexpr unsigned long long DEADLINE_TICKS = 50000000ull;

constexpr int PBLK = 272;
constexpr int PWAVE = 16 * PBLK;
constexpr int PART_BYTES = 8 * PWAVE;
constexpr int STA_OFF = 0;
constexpr int STB_OFF = 72 * 1024;
constexpr int CTL_OFF = STB_OFF + 8 * 8192;
constexpr int LDS_BYTES = CTL_OFF + 256;
static_assert(2 * PART_BYTES <= STB_OFF, "LDS map");

constexpr int AUX_SYS = 17;
constexpr int AUX_NT = 2;

constexpr int IDLE_XCD = 1;
constexpr signed char PEER_OF_XCD[NR][8] = {
    {6, -1, 2, 3, 1, 5, 4, 7}, {7, -1, 4, 5, 0, 3, 2, 6}, {3, -1, 0, 6, 4, 5, 1, 7}, {2, -1, 0, 6, 1, 5, 4, 7},
    {5, -1, 1, 7, 0, 2, 3, 6}, {4, -1, 1, 7, 2, 3, 0, 6}, {0, -1, 2, 3, 4, 7, 1, 5}, {1, -1, 4, 5, 3, 6, 0, 2}};
constexpr bool peers_ok() {
  for (int r = 0; r < NR; ++r) {
    unsigned m = 1u << r;
    for (int x = 0; x < 8; ++x) { if (x == IDLE_XCD) continue; if (PEER_OF_XCD[r][x] < 0) return false; m |= 1u << PEER_OF_XCD[r][x]; }
    if (m != 0xffu) return false;
  }
  return true;
}
static_assert(peers_ok(), "every rank pushes to each of its seven peers exactly once");

struct Params {
  const char* x;
  const char* w;
  const float* sx;
  const float* sw;
  float* out;
  char* my;
  char* peer_ws[NR];
  int rank;
};

template <int N>
__device__ __forceinline__ void wait_vm() {
  static_assert(N >= 0 && N < 64, "vmcnt range");
  asm volatile("s_waitcnt vmcnt(%0)" ::"n"(N) : "memory");
}
__device__ __forceinline__ void wait_lgkm0() { asm volatile("s_waitcnt lgkmcnt(0)" ::: "memory"); }
__device__ __forceinline__ void cfence() { asm volatile("" ::: "memory"); }
__device__ __forceinline__ unsigned lds_addr(const LAS void* p) { return (unsigned)(unsigned long long)p; }

__device__ __forceinline__ u32x4 make_srd(const void* base, unsigned bytes) {
  const unsigned long long a = (unsigned long long)base;
  u32x4 r;
  r.x = (unsigned)__builtin_amdgcn_readfirstlane((unsigned)a);
  r.y = (unsigned)__builtin_amdgcn_readfirstlane((unsigned)(a >> 32) & 0xffffu);
  r.z = bytes;
  r.w = 0x00020000u;
  return r;
}
__device__ __forceinline__ void dma2(u32x4 srd, unsigned lds, unsigned voff, unsigned s0, unsigned s1) {
  asm volatile(
      "s_mov_b32 m0, %0\n\ts_nop 3\n\t"
      "buffer_load_dwordx4 %1, %2, %3 offen lds\n\t"
      "s_add_u32 m0, m0, 1024\n\ts_nop 0\n\t"
      "buffer_load_dwordx4 %1, %2, %4 offen lds"
      ::"s"(lds), "v"(voff), "s"(srd), "s"(s0), "s"(s1)
      : "memory", "scc");
}
__device__ __forceinline__ void dma_w(u32x4 srd, unsigned lds, unsigned v_even, unsigned v_odd, unsigned soff) {
  asm volatile(
      "s_mov_b32 m0, %0\n\ts_nop 3\n\t"
      "buffer_load_dwordx4 %1, %3, %4 offen lds\n\t"
      "s_add_u32 m0, m0, 1024\n\ts_nop 0\n\t"
      "buffer_load_dwordx4 %2, %3, %4 offen lds\n\t"
      "s_add_u32 m0, m0, 1024\n\ts_nop 0\n\t"
      "buffer_load_dwordx4 %1, %3, %5 offen lds\n\t"
      "s_add_u32 m0, m0, 1024\n\ts_nop 0\n\t"
      "buffer_load_dwordx4 %2, %3, %5 offen lds"
      ::"s"(lds), "v"(v_even), "v"(v_odd), "s"(srd), "s"(soff), "s"(soff + 32u * NSH)
      : "memory", "scc");
}

__device__ __forceinline__ unsigned strip_src(int lane) { return (lane >> 3) * KDIM + ((lane & 7) ^ (lane >> 3)) * 16; }
__device__ __forceinline__ unsigned strip_frag(int lane, int f) { return (lane & 15) * 128 + ((4 * f + (lane >> 4)) ^ (lane & 7)) * 16; }

struct CommCtx {
  u32x4 x_srd;
  __amdgpu_buffer_rsrc_t dr;
  unsigned xoff, doff;
  unsigned ring_a;
  const LAS char* rd0;
  const LAS char* rd1;
  unsigned* flag;
  unsigned call;
};

constexpr int CD = 3;
constexpr int CRING = 8;
static_assert(CD >= 3 && CD < CRING && 8 * CRING * 2048 <= LDS_BYTES, "comm ring");

__device__ __forceinline__ void comm_load(const CommCtx& c, int rb) {
  dma2(c.x_srd, c.ring_a + (rb & (CRING - 1)) * 2048, c.xoff, rb * RB_BYTES, rb * RB_BYTES + 8u * KDIM);
}
__device__ __forceinline__ void comm_read(const CommCtx& c, int rb, u32x4 (&v)[2]) {
  v[0] = *(const LAS u32x4*)(c.rd0 + (rb & (CRING - 1)) * 2048);
  v[1] = *(const LAS u32x4*)(c.rd1 + (rb & (CRING - 1)) * 2048);
}
__device__ __forceinline__ void comm_store(const CommCtx& c, int rb, int f, u32x4 v) {
  __builtin_amdgcn_raw_buffer_store_b128(v, c.dr, c.doff + f * 1024 + rb * RB_BYTES, 0, AUX_SYS);
  cfence();
}
__device__ __forceinline__ void comm_signal(const CommCtx& c, int r) {
  __builtin_amdgcn_s_barrier();
  cfence();
  if ((int)threadIdx.x == (r & 7) * 64) __hip_atomic_store(c.flag + r, c.call, __ATOMIC_RELAXED, __HIP_MEMORY_SCOPE_SYSTEM);
  cfence();
}
__device__ __forceinline__ void comm_step(const CommCtx& c, int r, u32x4 (&cur)[2]) {
  wait_vm<3>();
  comm_store(c, r, 0, cur[0]);
  wait_vm<3>();
  comm_store(c, r, 1, cur[1]);
  comm_load(c, r + CD);
  comm_read(c, r + 1, cur);
  comm_signal(c, r - 1);
}

template <int V> using ic = std::integral_constant<int, V>;
enum { FILL_NONE = 0, FILL_LOCAL = 1, FILL_REMOTE = 2 };

__global__ void __launch_bounds__(512, 2) ag_gemm_kernel(Params p) {
  __shared__ __attribute__((aligned(16))) char smem_raw[LDS_BYTES];
  LAS char* smem = (LAS char*)smem_raw;
  const int b = blockIdx.x, xcd = b & 7, slot = b >> 3;
  const int wave = __builtin_amdgcn_readfirstlane(threadIdx.x >> 6), lane = threadIdx.x & 63;
  const int rank = p.rank & (NR - 1);
  char* my = p.my;
  unsigned* ctr = (unsigned*)(my + CTR_OFF + (size_t)b * 64);
  const char* const px = p.x;
  const char* const pwt = p.w;
  asm volatile("" ::"s"(px), "s"(pwt), "s"(my), "s"(rank));
  const u32x4 x_srd = make_srd(px, SHARD_BYTES);

  if (__builtin_expect(slot < 4, 1)) {
    const int q = slot;
    if (xcd == IDLE_XCD) return;
    const unsigned strip = q * 8 + wave;
    CommCtx c;
    c.x_srd = x_srd;
    c.xoff = strip_src(lane) + strip * 128;
    LAS char* ring = smem + wave * (CRING * 2048);
    c.ring_a = lds_addr(ring);
    comm_load(c, 0);
    comm_load(c, 1);
    c.rd0 = ring + strip_frag(lane, 0);
    c.rd1 = ring + strip_frag(lane, 1);
    unsigned call;
    asm volatile("s_load_dword %0, %1, 0x0\n\ts_waitcnt lgkmcnt(0)" : "=s"(call) : "s"(ctr) : "memory");
    call += 1u;
    char* const pw = p.peer_ws[xcd];
    c.dr = __builtin_amdgcn_make_buffer_rsrc((void*)(pw + STAGE_OFF + (size_t)((call & 1u) * NR + rank) * SHARD_BYTES), 0,
                                             (int)SHARD_BYTES, 0x00020000);
    c.doff = lane * 16 + strip * 2048;
    c.flag = (unsigned*)(pw + FLAG_OFF + (size_t)(rank * 4 + q) * 256);
    c.call = call;
    u32x4 v[2];
    wait_vm<2>();
    comm_read(c, 0, v);
    cfence();
#pragma unroll
    for (int r = 2; r < CD; ++r) comm_load(c, r);
    comm_store(c, 0, 0, v[0]);
    comm_store(c, 0, 1, v[1]);
    comm_load(c, CD);
    wait_vm<2 * (CD - 2) + 4>();
    comm_read(c, 1, v);
#pragma nounroll
    for (int r = 1; r < NRB; ++r) comm_step(c, r, v);
    wait_vm<0>();
    comm_signal(c, NRB - 1);
    if (threadIdx.x == 0) __hip_atomic_store(ctr, call, __ATOMIC_RELAXED, __HIP_MEMORY_SCOPE_AGENT);
    return;
  }

  const int u_ = (xcd >> 2) * 28 + slot - 4;
  const int panel = (xcd & 3) * 8 + (u_ & 7);
  const int si = u_ >> 3;
  const int src = (rank + 1 + si) & (NR - 1);
  const int lc = lane & 15, lg = lane >> 4;
  const int rot = (panel + 2 * si) & 7;
  const int kr = wave ^ ((rot & 1) << 2);
  const int ks0 = rot & 6;
  LAS float* lscale = (LAS float*)(smem + CTL_OFF);

  LAS char* const sta = smem + STA_OFF + wave * 8192;
  LAS char* const stb = smem + STB_OFF + wave * 8192;
  const unsigned xl_off = strip_src(lane) + kr * 512;
  auto local_load = [&](int i, int h, LAS char* dst) {
    const unsigned so = (unsigned)(si + 7 * i) * RB_BYTES;
#pragma unroll
    for (int s = 0; s < 2; ++s) {
      const unsigned o = so + (unsigned)((4 * h + 2 * s + ks0) & 7) * 64u;
      dma2(x_srd, lds_addr(dst) + s * 2048, xl_off, o, o + 8u * KDIM);
    }
  };
  const unsigned xf0 = strip_frag(lane, 0), xf1 = strip_frag(lane, 1);
  auto local_take = [&](i32x4 (&xf)[8], int h, const LAS char* s) {
    xf[4 * h + 0] = *(const LAS i32x4*)(s + xf0);
    xf[4 * h + 1] = *(const LAS i32x4*)(s + xf1);
    xf[4 * h + 2] = *(const LAS i32x4*)(s + 2048 + xf0);
    xf[4 * h + 3] = *(const LAS i32x4*)(s + 2048 + xf1);
  };

  i32x4 wf[8][4], xa[8], xb[8], la[2][4];
  float sxv, swv;
  unsigned ctr_val;
  {
    const u32x4 w_srd = make_srd(pwt, W_BYTES);
    const unsigned pc = lane & 3;
    const unsigned vb = (unsigned)(kr * 512 + (lane >> 2)) * NSH + panel * 64;
    const unsigned v_even = vb + (pc ^ (lg & 1)) * 16, v_odd = vb + 16 * NSH + (pc ^ ((lg & 1) | 2)) * 16;
    auto ring = [&](int c) -> LAS char* { return ((c & 2) ? sta : stb) + (c & 1) * 4096; };
    auto w_load = [&](int t) { dma_w(w_srd, lds_addr(ring(t)), v_even, v_odd, (unsigned)((t + ks0) & 7) * 64u * NSH); };
    const unsigned q = lc >> 1, rsw = ((q >> 2) & 1) | ((lg & 1) << 1);
    const unsigned wro = (16 * lg + q) * 64 + 8 * (lane & 1);
    auto w_take = [&](int t) {
#pragma unroll
      for (int cg = 0; cg < 4; ++cg) {
        const LAS char* a = ring(t) + wro + (cg ^ rsw) * 16;
        const i32x2 lo = __builtin_amdgcn_ds_read_tr8_b64_v2i32((LAS i32x2*)a);
        const i32x2 hi = __builtin_amdgcn_ds_read_tr8_b64_v2i32((LAS i32x2*)(a + 512));
        wf[t][cg] = (i32x4){lo[0], lo[1], hi[0], hi[1]};
      }
      wait_lgkm0();
      asm volatile("" : "+v"(wf[t][0]), "+v"(wf[t][1]), "+v"(wf[t][2]), "+v"(wf[t][3]));
    };
#pragma unroll
    for (int c = 0; c < 4; ++c) local_load(c >> 1, c & 1, ring(c));
    asm volatile("s_load_dword %0, %3, 0x0\n\ts_load_dword %1, %4, 0x0\n\ts_load_dword %2, %5, 0x0\n\ts_waitcnt lgkmcnt(0)"
                 : "=&s"(sxv), "=&s"(swv), "=&s"(ctr_val) : "s"(p.sx), "s"(p.sw), "s"(ctr) : "memory");
#pragma unroll
    for (int c = 0; c < 4; ++c) {
      wait_vm<12>();
      i32x4(&xf)[8] = (c >> 1) ? xb : xa;
      local_take(xf, c & 1, ring(c));
      wait_lgkm0();
      const int h = 4 * (c & 1);
      asm volatile("" : "+v"(xf[h]), "+v"(xf[h + 1]), "+v"(xf[h + 2]), "+v"(xf[h + 3]));
      w_load(c);
    }
    auto mul01 = [&](int t) {
#pragma unroll
      for (int g = 0; g < 4; ++g) {
        if (t == 0) {
          asm volatile("v_mfma_i32_16x16x64_i8 %0, %1, %2, 0" : "=&v"(la[0][g]) : "v"(wf[t][g]), "v"(xa[t]));
          asm volatile("v_mfma_i32_16x16x64_i8 %0, %1, %2, 0" : "=&v"(la[1][g]) : "v"(wf[t][g]), "v"(xb[t]));
        } else {
          asm volatile("v_mfma_i32_16x16x64_i8 %0, %1, %2, %0" : "+v"(la[0][g]) : "v"(wf[t][g]), "v"(xa[t]));
          asm volatile("v_mfma_i32_16x16x64_i8 %0, %1, %2, %0" : "+v"(la[1][g]) : "v"(wf[t][g]), "v"(xb[t]));
        }
      }
    };
#pragma unroll
    for (int t = 0; t < 4; ++t) { wait_vm<12>(); w_take(t); w_load(t + 4); mul01(t); }
    wait_vm<12>(); w_take(4); local_load(2, 0, stb); mul01(4);
    wait_vm<12>(); w_take(5); local_load(2, 1, stb + 4096); mul01(5);
    wait_vm<12>(); w_take(6); mul01(6);
    wait_vm<8>();  w_take(7); mul01(7);
    asm volatile("s_nop 7\n\ts_nop 7" ::: "memory");
  }
  if (threadIdx.x == 0) *lscale = sxv * swv;
  const unsigned call = ctr_val + 1u;

  const __amdgpu_buffer_rsrc_t sr = __builtin_amdgcn_make_buffer_rsrc(
      (void*)(my + STAGE_OFF + (size_t)((call & 1u) * NR + src) * SHARD_BYTES), 0, (int)SHARD_BYTES, 0x00020000);
  const __amdgpu_buffer_rsrc_t orr = __builtin_amdgcn_make_buffer_rsrc((void*)p.out, 0, (int)OUT_BYTES, 0x00020000);
  const unsigned* flag = (const unsigned*)(my + FLAG_OFF + (size_t)(src * 4 + (kr >> 1)) * 256) + (lane & 31);
  auto poll = [&]() -> unsigned { return __hip_atomic_load(flag, __ATOMIC_RELAXED, __HIP_MEMORY_SCOPE_SYSTEM); };
  auto seen = [&](unsigned v) -> unsigned { return (unsigned)__ballot((int)(v - call) >= 0); };
  unsigned* tmo = (unsigned*)(my + TMO_OFF);
  const unsigned voff_s = lane * 16 + kr * 8192;
  const unsigned wr_off = wave * PWAVE + lg * PBLK + lc * 16;
  const unsigned rd_off = ((lane & 31) >> 1) * PBLK + (2 * wave + (lane >> 5)) * 16 + (lane & 1) * 8;
  const unsigned voff_o = (unsigned)((2 * wave + (lane >> 5)) * NSH + panel * 64 + (lane & 31) * 2) * 4u;

  unsigned ready = 0u;
  auto wait_rb = [&](int rb) {
    if (__builtin_expect(ready >> rb & 1u, 1)) return;
    const unsigned long long t0 = __builtin_amdgcn_s_memrealtime();
    bool gave_up = false;
    for (unsigned spins = 1;; ++spins) {
      ready |= seen(poll());
      if (ready >> rb & 1u) break;
      if ((spins & 31u) == 0u) {
        const unsigned tv = (unsigned)__builtin_amdgcn_readfirstlane(__hip_atomic_load(tmo, __ATOMIC_RELAXED, __HIP_MEMORY_SCOPE_AGENT));
        if (tv == call || __builtin_amdgcn_s_memrealtime() - t0 > DEADLINE_TICKS) { gave_up = true; break; }
      }
      __builtin_amdgcn_s_sleep(1);
    }
    cfence();
    if (gave_up) {
      __hip_atomic_store(tmo, call, __ATOMIC_RELAXED, __HIP_MEMORY_SCOPE_AGENT);
      __hip_atomic_store(lscale, __builtin_nanf(""), __ATOMIC_RELAXED, __HIP_MEMORY_SCOPE_WORKGROUP);
      ready = ~0u;
    }
    __builtin_amdgcn_s_waitcnt(0x0070);
  };
  auto load_frag = [&](int rb, int t) -> i32x4 {
    return __builtin_bit_cast(i32x4, __builtin_amdgcn_raw_buffer_load_b128(
        sr, voff_s + (t & 1) * 1024, (unsigned)rb * RB_BYTES + (unsigned)(((t & 6) + ks0) & 7) * 1024u, AUX_SYS));
  };

  i32x2 part[8];
  float sc;
  auto read_part = [&](int j, int w) { part[w] = *(const LAS i32x2*)(smem + (j & 1) * PART_BYTES + rd_off + w * PWAVE); };
  auto read_scale = [&]() { sc = __hip_atomic_load(lscale, __ATOMIC_RELAXED, __HIP_MEMORY_SCOPE_WORKGROUP); };
  auto reduce_read = [&](int j) {
#pragma unroll
    for (int w = 0; w < 8; ++w) read_part(j, w);
    read_scale();
  };
  const unsigned grb_l = (unsigned)(rank * NRB + si), grb_l4 = si + 28 < NRB ? grb_l + 28u : 256u, grb_r = (unsigned)(src * NRB - NLOC);
  auto row_block = [&](int j) -> unsigned {
    const unsigned l = j < 0 ? 256u : (j == 4 ? grb_l4 : grb_l + 7u * (unsigned)j);
    return j >= NLOC ? grb_r + (unsigned)j : l;
  };
  auto put = [&](i32x2 s, unsigned grb) {
    f32x2 y;
    y[0] = (float)s[0] * sc;
    y[1] = (float)s[1] * sc;
    __builtin_amdgcn_raw_buffer_store_b64(__builtin_bit_cast(u32x2, y), orr, voff_o + grb * (16u * NSH * 4u), 0, AUX_NT);
  };
  auto reduce_store = [&](int j) {
    put(((part[0] + part[1]) + (part[2] + part[3])) + ((part[4] + part[5]) + (part[6] + part[7])), row_block(j));
  };

  cfence();
  __builtin_amdgcn_s_barrier();
  cfence();
  if (threadIdx.x == 0) __hip_atomic_store(ctr, call, __ATOMIC_RELAXED, __HIP_MEMORY_SCOPE_AGENT);
#pragma unroll
  for (int i = 0; i < 2; ++i)
#pragma unroll
    for (int g = 0; g < 4; ++g) *(LAS i32x4*)(smem + i * PART_BYTES + wr_off + g * 4 * PBLK) = la[i][g];
  wait_vm<0>();
  local_take(xa, 0, stb);
  local_take(xa, 1, stb + 4096);
  wait_lgkm0();
  asm volatile("" : "+v"(xa[0]), "+v"(xa[1]), "+v"(xa[2]), "+v"(xa[3]), "+v"(xa[4]), "+v"(xa[5]), "+v"(xa[6]), "+v"(xa[7]));
  local_load(3, 0, stb);
  local_load(3, 1, stb + 4096);
  __builtin_amdgcn_s_barrier();
  cfence();
  reduce_read(0);
  reduce_store(0);
  cfence();
  reduce_read(1);
  reduce_store(1);
  cfence();
  __builtin_amdgcn_s_barrier();
  cfence();

  const int rlag = wave < 4 ? 1 : 0;
  unsigned pollv = call - 1u;
  auto step = [&](i32x4 (&xf)[8], i32x4 (&xo)[8], int j, auto fill) {
    constexpr int FILL = decltype(fill)::value;
    LAS char* pwr = smem + (j & 1) * PART_BYTES + wr_off;
    i32x4 acc[4];
#pragma unroll
    for (int g = 0; g < 4; ++g) {
      acc[g] = (i32x4){0, 0, 0, 0};
#pragma unroll
      for (int t = 0; t < 8; ++t) {
        acc[g] = __builtin_amdgcn_mfma_i32_16x16x64_i8(wf[t][g], xf[t], acc[g], 0, 0, 0);
        if (t == 7) __builtin_amdgcn_sched_barrier(0);
        if (g > 0 && t == 6) {
          __builtin_amdgcn_sched_barrier(0);
          *(LAS i32x4*)(pwr + (g - 1) * 4 * PBLK) = acc[g - 1];
          __builtin_amdgcn_sched_barrier(0);
        }
      }
    }
    if constexpr (FILL == FILL_REMOTE) ready |= seen(pollv);
    unsigned grb = row_block(j - rlag);
    asm volatile("" : "+s"(grb), "+s"(ready));
    *(LAS i32x4*)(pwr + 3 * 4 * PBLK) = acc[3];
    if (wave >= 4) wait_lgkm0();
    cfence();
    __builtin_amdgcn_s_barrier();
    __builtin_amdgcn_sched_barrier(0);
    cfence();
    if constexpr (FILL != FILL_REMOTE) reduce_read(j - rlag);
    cfence();
    if constexpr (FILL == FILL_REMOTE) {
      if (j == 3) {
        wait_vm<2>();
        local_take(xo, 0, stb);
        local_take(xo, 1, stb + 4096);
      }
      const int rb = j + 2 - NLOC;
      wait_rb(rb < NRB ? rb : 0);
      cfence();
      pollv = poll();
      cfence();
      i32x2 s0, s1;
#pragma unroll
      for (int t = 0; t < 8; ++t) {
        xf[t] = load_frag(rb, t);
        if (t < 2) { read_part(j - rlag, 3 * t); read_part(j - rlag, 3 * t + 1); read_part(j - rlag, 3 * t + 2); }
        if (t == 2) { read_part(j - rlag, 6); read_part(j - rlag, 7); read_scale(); }
        if (t == 4) { s0 = part[0] + part[1] + part[2]; s1 = part[3] + part[4] + part[5]; asm volatile("" : "+v"(s0), "+v"(s1)); }
        if (t == 5) { s0 = (s0 + s1) + (part[6] + part[7]); asm volatile("" : "+v"(s0)); }
        if (t == 6) put(s0, grb);
        __builtin_amdgcn_sched_barrier(0);
      }
    } else if constexpr (FILL == FILL_LOCAL) {
      wait_vm<2>();
      local_take(xo, 0, stb);
      local_take(xo, 1, stb + 4096);
      wait_lgkm0();
      asm volatile("" : "+v"(xo[0]), "+v"(xo[1]), "+v"(xo[2]), "+v"(xo[3]), "+v"(xo[4]), "+v"(xo[5]), "+v"(xo[6]), "+v"(xo[7]));
      local_load(4, 0, stb);
      local_load(4, 1, stb + 4096);
      pollv = poll();
      cfence();
      reduce_store(rlag ? -1 : j);
    } else {
      reduce_store(j - rlag);
      if (wave >= 4) return;
    }
    wait_lgkm0();
    __builtin_amdgcn_s_barrier();
    __builtin_amdgcn_sched_barrier(0);
    cfence();
  };

  if (wave >= 4) __builtin_amdgcn_s_barrier();
  step(xa, xb, 2, ic<FILL_LOCAL>{});
  step(xb, xa, 3, ic<FILL_REMOTE>{});
#pragma nounroll
  for (int j = 4; j < NTOT - 1; j += 2) {
    step(xa, xb, j, ic<FILL_REMOTE>{});
    step(xb, xa, j + 1, ic<FILL_REMOTE>{});
  }
  step(xa, xb, NTOT - 1, ic<FILL_NONE>{});
  if (wave < 4) {
    reduce_read(NTOT - 1);
    reduce_store(NTOT - 1);
  }
}

}

extern "C" void kernel_launch(int rank, int nranks, ncclComm_t comm,
                              hipStream_t stream,
                              void* const* d_in, const int* in_sizes, int n_in,
                              void* d_out, int out_size,
                              const int* in_dtypes, int out_dtype,
                              void* const* d_out_all,
                              void* const* d_ws_all, size_t ws_bytes) {
  (void)comm; (void)in_dtypes; (void)out_dtype; (void)d_out_all;
  if (nranks != agk::NR || (unsigned)rank >= (unsigned)agk::NR || n_in < 4 || ws_bytes < agk::WS_NEED ||
      in_sizes[0] != agk::MSH * agk::KDIM || in_sizes[1] != agk::KDIM * agk::NSH ||
      out_size != agk::NR * agk::MSH * agk::NSH) {
    fprintf(stderr, "kernel_launch: unexpected problem shape\n");
    return;
  }
  agk::Params p;
  p.x = (const char*)d_in[0];
  p.w = (const char*)d_in[1];
  p.sx = (const float*)d_in[2];
  p.sw = (const float*)d_in[3];
  p.out = (float*)d_out;
  p.my = (char*)d_ws_all[rank];
  for (int x = 0; x < agk::NR; ++x) {
    const int peer = agk::PEER_OF_XCD[rank][x];
    p.peer_ws[x] = (char*)d_ws_all[peer < 0 ? rank : peer];
  }
  p.rank = rank;
  hipLaunchKernelGGL(agk::ag_gemm_kernel, dim3(256), dim3(512), 0, stream, p);
  HIP_CHECK(hipGetLastError());
}
```
